```python
import numpy as np

kernel_name = "gpu_a2a_gemm_m4096_k4096_n32768_bf16_relu"


ABSMAX_RTOL = 0.008

COMPUTE_DTYPE = "bfloat16"
EPILOGUE = "relu"
IN_QUANT = None
OUT_QUANT = None
IN_DTYPES = {"x": "bfloat16", "w_mat": "bfloat16"}
WORLD_SIZE = 8

def _to_bf16(x):
    u = np.ascontiguousarray(x, dtype=np.float32).view(np.uint32)
    return ((u + 0x7FFF + ((u >> 16) & 1)) & 0xFFFF0000).view(np.float32)

def setup_inputs():

    x = _to_bf16(np.random.randn(4096, 4096).astype(np.float32))
    w_mat = _to_bf16((np.random.randn(4096, 32768) / np.sqrt(4096)).astype(np.float32))
    return {"x": x, "w_mat": w_mat}

def _epilogue(y):
    return np.maximum(y, 0.0).astype(np.float32)

def reference(x, w_mat):
    y = (x @ w_mat).astype(np.float32)
    return _epilogue(y)
```

```cpp
#include <hip/hip_runtime.h>
#include <rccl/rccl.h>
#include <cstdio>
#include <cstdint>

#define HIP_CHECK(expr) do { hipError_t _e = (expr); if (_e != hipSuccess) { \
    fprintf(stderr, "HIP error %s at %s:%d\n", hipGetErrorString(_e), __FILE__, __LINE__); \
    return; } } while (0)

#define LAS __attribute__((address_space(3)))
#define GAS __attribute__((address_space(1)))
typedef short s16x8 __attribute__((ext_vector_type(8)));
typedef short s16x4 __attribute__((ext_vector_type(4)));
typedef short s16x2 __attribute__((ext_vector_type(2)));
typedef __bf16 bf16x8 __attribute__((ext_vector_type(8)));
typedef float f32x4 __attribute__((ext_vector_type(4)));
typedef unsigned u32x4 __attribute__((ext_vector_type(4)));

constexpr int NR = 8;
constexpr int MLOC = 512;
constexpr int KDIM = 4096;
constexpr int NDIM = 32768;
constexpr int KC = KDIM / NR;
constexpr unsigned CHUNK_BYTES = MLOC * KC * 2;
constexpr int HTB = 16384;
constexpr int LDS_BYTES = 8 * HTB;
constexpr unsigned W_BYTES = (unsigned)KDIM * NDIM * 2u;
constexpr unsigned WROW = NDIM * 2;
constexpr unsigned AROW = KC * 2;
constexpr int GRID = 256;
constexpr size_t WS_CNT = 0;
constexpr size_t WS_TMO = 128;
constexpr size_t WS_FLAGS = 4096;
constexpr size_t WS_STAGE = 1u << 20;
constexpr size_t SLOT_BYTES = (size_t)NR * CHUNK_BYTES;
constexpr size_t WS_NEED = WS_STAGE + 2 * SLOT_BYTES;
constexpr unsigned long long DEADLINE_TICKS = 40000000ull;

struct Params {
    const char* x;
    const char* w;
    char* out;
    char* ws[NR];
    int rank;
};

__device__ __forceinline__ unsigned cvt_pk_bf16(float lo, float hi) {
    unsigned r; asm("v_cvt_pk_bf16_f32 %0, %1, %2" : "=v"(r) : "v"(lo), "v"(hi)); return r;
}
__device__ __forceinline__ unsigned relu_pk(unsigned v) {
    s16x2 a = __builtin_bit_cast(s16x2, v);
    s16x2 z = {0, 0};
    return __builtin_bit_cast(unsigned, __builtin_elementwise_max(a, z));
}

template <class T> __device__ __forceinline__ T* uni(T* ptr) {
    const unsigned long long v = (unsigned long long)ptr;
    const unsigned lo = __builtin_amdgcn_readfirstlane((unsigned)v), hi = __builtin_amdgcn_readfirstlane((unsigned)(v >> 32));
    return (T*)(((unsigned long long)hi << 32) | lo);
}

__device__ __forceinline__ void pair_map(int u, int rank, int& chunk, int& sub) {
    if (u < 4) { chunk = rank; sub = u; }
    else if (u < 11) { chunk = (rank + 1 + (u - 4)) & 7; sub = 0; }
    else { int v = u - 11; int d = (v * 43) >> 7; chunk = (rank + 1 + d) & 7; sub = 1 + (v - d * 3); }
}

__global__ __launch_bounds__(512, 2) void a2a_gemm_kernel(Params p) {
    extern __shared__ __attribute__((aligned(16))) char smem_[];
    LAS char* lds = (LAS char*)smem_;
    const int tid = threadIdx.x;
    const int wid = __builtin_amdgcn_readfirstlane(tid >> 6);
    const int lane = tid & 63, wr = wid >> 2, wc = wid & 3, fr = lane & 15, fq = lane >> 4;
    const int bid = blockIdx.x;
    const int xcd = bid & 7, jj = bid >> 3;
    const int pm = jj & 1, pn = xcd * 16 + (jj >> 1);
    const int rank = p.rank;
    char* const myws = p.ws[rank];

    const unsigned cnt = (unsigned)__builtin_amdgcn_readfirstlane(
        __hip_atomic_load((GAS unsigned*)(myws + WS_CNT), __ATOMIC_RELAXED, __HIP_MEMORY_SCOPE_AGENT));
    const unsigned call = cnt / GRID + 1u;
    const size_t slot_off = WS_STAGE + (size_t)(call & 1u) * SLOT_BYTES;

    {
#pragma unroll
        for (int round = 0; round < 2; ++round) {
            const int i = round * 2048 + bid * 8 + wid;
            if (i < 3584) {
                const int sd = i >> 7, rg = i & 127;
                const int sub = sd / 7, d = sd - sub * 7;
                const unsigned dest = (unsigned)(rank + 1 + d) & 7u;
                const unsigned row = (unsigned)rg * 4u + (unsigned)(lane >> 4);
                const unsigned coff = (unsigned)sub * 256u + (unsigned)(lane & 15) * 16u;
                const u32x4 v = *(const u32x4*)(p.x + (size_t)(dest * MLOC + row) * AROW + coff);
                *(u32x4*)(p.ws[dest] + slot_off + (size_t)rank * CHUNK_BYTES + (size_t)row * AROW + coff) = v;
            }
        }
        asm volatile("s_waitcnt vmcnt(0)" ::: "memory");
        __syncthreads();
        if (tid == 0) {
            __builtin_amdgcn_fence(__ATOMIC_RELEASE, "");
            asm volatile("s_waitcnt vmcnt(0)" ::: "memory");
#pragma unroll
            for (int round = 0; round < 2; ++round) {
                const int i = round * 2048 + bid * 8;
                if (i < 3584) {
                    const int sd = i >> 7, sub = sd / 7, d = sd - sub * 7;
                    const unsigned dest = (unsigned)(rank + 1 + d) & 7u;
                    const int slot = (i & 127) >> 3;
                    __hip_atomic_store((GAS unsigned*)(p.ws[dest] + WS_FLAGS + (size_t)(rank * 4 + sub) * 128 + slot * 4), call,
                                       __ATOMIC_RELAXED, __HIP_MEMORY_SCOPE_SYSTEM);
                }
            }
        }
    }

    const unsigned voffA = (unsigned)(lane >> 3) * AROW + (unsigned)(((lane & 7) ^ (lane >> 3)) * 16);
    const unsigned voffB = (unsigned)(((lane >> 3) & 1) * 8 + (lane >> 4)) * WROW + (unsigned)(((lane & 7) ^ ((lane >> 4) << 1)) * 16);
    const unsigned sA_w = (unsigned)(pm * 256 + wid * 8) * AROW;
    const unsigned sB_w = (unsigned)(((wid >> 2) & 1) * 16 + ((wid >> 1) & 1) * 4) * WROW + (unsigned)(wid & 1) * 128u + (unsigned)pn * 512u;
    const unsigned ldsw = (unsigned)wid * 1024u;
    const int arow = wr * 64 + fr;
    const unsigned aoff0 = (unsigned)arow * 128u + (unsigned)((fq ^ (fr & 7)) * 16);
    const unsigned aoff1 = aoff0 ^ 64u;
    const int q = (lane & 15) >> 2, pp = lane & 3;
    const unsigned bcom = (unsigned)(fq >> 1) * 4096u + (unsigned)(wc >> 1) * 1024u + (unsigned)q * 256u + (unsigned)(fq & 1) * 128u + (unsigned)pp * 8u;
    const unsigned ldsbase = (unsigned)(size_t)lds;
    const unsigned boff0 = ldsbase + 4 * HTB + bcom + (unsigned)(((2 * (wc & 1) + 0) ^ q) * 32);
    const unsigned boff1 = ldsbase + 4 * HTB + bcom + (unsigned)(((2 * (wc & 1) + 1) ^ q) * 32);

#define SA(b, h) (((b) * 2 + (h)) * HTB)
#define SB(b, h) (((b) * 2 + (h)) * HTB)
#define STAGE_A(b, h, rs, so) do { \
        __builtin_amdgcn_raw_ptr_buffer_load_lds(rs, (LAS void*)(lds + SA(b, h) + ldsw), 16, voffA, (so) + sA_w + (h) * 128u * AROW, 0, 0); \
        __builtin_amdgcn_raw_ptr_buffer_load_lds(rs, (LAS void*)(lds + SA(b, h) + ldsw + 8192), 16, voffA, (so) + sA_w + (h) * 128u * AROW + 64u * AROW, 0, 0); } while (0)
#define STAGE_B(b, h, rs, so) do { \
        __builtin_amdgcn_raw_ptr_buffer_load_lds(rs, (LAS void*)(lds + 4 * HTB + SB(b, h) + ldsw), 16, voffB, (so) + sB_w + (h) * 256u, 0, 0); \
        __builtin_amdgcn_raw_ptr_buffer_load_lds(rs, (LAS void*)(lds + 4 * HTB + SB(b, h) + ldsw + 8192), 16, voffB, (so) + sB_w + (h) * 256u + 32u * WROW, 0, 0); } while (0)
#define LDA(dst, b, h) do { _Pragma("unroll") for (int m = 0; m < 4; ++m) { \
        dst[m][0] = *(const LAS s16x8*)(lds + SA(b, h) + aoff0 + m * 2048); \
        dst[m][1] = *(const LAS s16x8*)(lds + SA(b, h) + aoff1 + m * 2048); } } while (0)
#define TRR(dst, base, off) asm volatile("ds_read_b64_tr_b16 %0, %1 offset:%2" : "=v"(dst) : "v"(base), "i"(off) : "memory")
#define LDB(dst, b, h) do { _Pragma("unroll") for (int k = 0; k < 2; ++k) { \
        { s16x4 lo, hi; TRR(lo, boff0, SB(b, h) + k * 8192); TRR(hi, boff0, SB(b, h) + k * 8192 + 2048); dst[0][k] = __builtin_shufflevector(lo, hi, 0, 1, 2, 3, 4, 5, 6, 7); } \
        { s16x4 lo, hi; TRR(lo, boff1, SB(b, h) + k * 8192); TRR(hi, boff1, SB(b, h) + k * 8192 + 2048); dst[1][k] = __builtin_shufflevector(lo, hi, 0, 1, 2, 3, 4, 5, 6, 7); } } } while (0)
#define MMA(ai, bj, At, Bt) do { __builtin_amdgcn_s_setprio(1); _Pragma("unroll") for (int m = 0; m < 4; ++m) _Pragma("unroll") for (int n = 0; n < 2; ++n) _Pragma("unroll") for (int k = 0; k < 2; ++k) \
        acc[ai][bj][m][n] = __builtin_amdgcn_mfma_f32_16x16x32_bf16(__builtin_bit_cast(bf16x8, Bt[n][k]), __builtin_bit_cast(bf16x8, At[m][k]), acc[ai][bj][m][n], 0, 0, 0); __builtin_amdgcn_s_setprio(0); } while (0)
#define WAIT_V(n) asm volatile("s_waitcnt vmcnt(" #n ")" ::: "memory")
#define WAIT_L(n) asm volatile("s_waitcnt lgkmcnt(" #n ")" ::: "memory")
#define BAR __builtin_amdgcn_s_barrier()
#define SCHED __builtin_amdgcn_sched_barrier(0)

    f32x4 acc[2][2][4][2];
#pragma unroll
    for (int a = 0; a < 2; ++a)
#pragma unroll
        for (int b = 0; b < 2; ++b)
#pragma unroll
            for (int m = 0; m < 4; ++m)
#pragma unroll
                for (int n = 0; n < 2; ++n) acc[a][b][m][n] = (f32x4){0.f, 0.f, 0.f, 0.f};
    s16x8 At[4][2], B0[2][2], B1[2][2];

    const auto rW = __builtin_amdgcn_make_buffer_rsrc((void*)p.w, 0, W_BYTES, 0x00020000);
    const auto rWz = __builtin_amdgcn_make_buffer_rsrc((void*)p.w, 0, 0, 0x00020000);
    const char* const xown = uni(p.x + (size_t)rank * CHUNK_BYTES);
    const char* const stage = uni(myws + slot_off);
    auto chunk_base = [&](int c) -> const char* { return c == rank ? xown : stage + (size_t)c * CHUNK_BYTES; };

    int ch, sub;
    pair_map(0, rank, ch, sub);
    auto rA = __builtin_amdgcn_make_buffer_rsrc((void*)chunk_base(ch), 0, CHUNK_BYTES, 0x00020000);
    unsigned soA = (unsigned)sub * 256u;
    unsigned soB = (unsigned)(ch * KC + sub * 128) * WROW;
    unsigned dead = 0;

    STAGE_B(0, 0, rW, soB); STAGE_A(0, 0, rA, soA); STAGE_B(0, 1, rW, soB); STAGE_A(0, 1, rA, soA);
    if (wr == 1) BAR;
    WAIT_V(4); BAR;
    STAGE_B(1, 0, rW, soB + 64u * WROW); STAGE_A(1, 0, rA, soA + 128u); STAGE_B(1, 1, rW, soB + 64u * WROW);
    WAIT_V(6); BAR;

    unsigned tmo_val = 0;
    for (int u = 0; u < 32; ++u) {
        const bool last = (u == 31);
        if (u == 3 || u == 10) {
            if (wid == 0) {
                const int nsub = (u == 3) ? 1 : 3, sub0 = (u == 3) ? 0 : 1;
                const unsigned long long t0 = __builtin_amdgcn_s_memrealtime();
                const unsigned src = (unsigned)(rank + 1 + (lane >> 3)) & 7u;
                const bool active = lane < 56;
                for (int s = 0; s < nsub; ++s) {
                    const GAS unsigned long long* fp = (const GAS unsigned long long*)(myws + WS_FLAGS + (size_t)(src * 4 + sub0 + s) * 128 + (lane & 7) * 8);
                    for (;;) {
                        const unsigned long long v = __hip_atomic_load(fp, __ATOMIC_RELAXED, __HIP_MEMORY_SCOPE_SYSTEM);
                        const bool ok = !active || ((unsigned)v >= call && (unsigned)(v >> 32) >= call);
                        if (__all(ok)) break;
                        const unsigned tm = __hip_atomic_load((GAS unsigned*)(myws + WS_TMO), __ATOMIC_RELAXED, __HIP_MEMORY_SCOPE_AGENT);
                        if (__builtin_amdgcn_readfirstlane(tm) == call || __builtin_amdgcn_s_memrealtime() - t0 > DEADLINE_TICKS) {
                            if (lane == 0) __hip_atomic_store((GAS unsigned*)(myws + WS_TMO), call, __ATOMIC_RELAXED, __HIP_MEMORY_SCOPE_AGENT);
                            break;
                        }
                        __builtin_amdgcn_s_sleep(2);
                    }
                }
                __builtin_amdgcn_fence(__ATOMIC_ACQUIRE, "");
                asm volatile("s_waitcnt vmcnt(0)" ::: "memory");
            }
            asm volatile("" ::: "memory"); BAR; asm volatile("" ::: "memory");
        }
        if (last) tmo_val = __hip_atomic_load((GAS unsigned*)(myws + WS_TMO), __ATOMIC_RELAXED, __HIP_MEMORY_SCOPE_AGENT);
        int ch2, sub2;
        pair_map(last ? 31 : u + 1, rank, ch2, sub2);
        const auto rA2 = __builtin_amdgcn_make_buffer_rsrc((void*)chunk_base(ch2), 0, last ? 0 : CHUNK_BYTES, 0x00020000);
        const auto rW2 = last ? rWz : rW;
        const unsigned soA2 = (unsigned)sub2 * 256u;
        const unsigned soB2 = (unsigned)(ch2 * KC + sub2 * 128) * WROW;
        LDB(B0, 0, 0); SCHED; LDA(At, 0, 0); STAGE_A(1, 1, rA, soA + 128u);
        WAIT_L(8); BAR; WAIT_L(0); SCHED; MMA(0, 0, At, B0); BAR; SCHED;
        LDB(B1, 0, 1); STAGE_B(0, 0, rW2, soB2);
        BAR; WAIT_L(0); SCHED; MMA(0, 1, At, B1); BAR;
        LDA(At, 0, 1); STAGE_A(0, 0, rA2, soA2);
        BAR; WAIT_L(0); SCHED; MMA(1, 0, At, B0); BAR; SCHED;
        STAGE_B(0, 1, rW2, soB2);
        WAIT_V(6); BAR; MMA(1, 1, At, B1); BAR;
        LDB(B0, 1, 0); SCHED; LDA(At, 1, 0); STAGE_A(0, 1, rA2, soA2);
        WAIT_L(8); BAR; WAIT_L(0); SCHED; MMA(0, 0, At, B0); BAR; SCHED;
        LDB(B1, 1, 1); STAGE_B(1, 0, rW2, soB2 + 64u * WROW);
        BAR; WAIT_L(0); SCHED; MMA(0, 1, At, B1); BAR;
        LDA(At, 1, 1); STAGE_A(1, 0, rA2, soA2 + 128u);
        BAR; WAIT_L(0); SCHED; MMA(1, 0, At, B0); BAR; SCHED;
        STAGE_B(1, 1, rW2, soB2 + 64u * WROW);
        WAIT_V(6); BAR; MMA(1, 1, At, B1); BAR;
        rA = rA2; soA = soA2; soB = soB2;
    }
    if (wr == 0) BAR;
    (void)dead;

    {
        int tz = threadIdx.x; asm volatile("" : "+v"(tz));
        const int l2 = tz & 63, fr2 = l2 & 15, fq2 = l2 >> 4;
        const bool bad = (tmo_val == call);
        char* obase = p.out + (size_t)(pm * 256 + wr * 64 + fr2) * WROW + (size_t)pn * 512 + wc * 64 + ((fq2 & 1) * 32 + (fq2 >> 1) * 16);
#pragma unroll
        for (int ai = 0; ai < 2; ++ai)
#pragma unroll
            for (int m = 0; m < 4; ++m) {
                char* rowp = obase + (size_t)(ai * 128 + m * 16) * WROW;
#pragma unroll
                for (int bj = 0; bj < 2; ++bj) {
                    const f32x4 v0 = acc[ai][bj][m][0], v1 = acc[ai][bj][m][1];
                    unsigned a0 = relu_pk(cvt_pk_bf16(v0[0], v0[1])), a1 = relu_pk(cvt_pk_bf16(v0[2], v0[3]));
                    unsigned b0 = relu_pk(cvt_pk_bf16(v1[0], v1[1])), b1 = relu_pk(cvt_pk_bf16(v1[2], v1[3]));
                    auto r0 = __builtin_amdgcn_permlane16_swap(a0, b0, false, false);
                    auto r1 = __builtin_amdgcn_permlane16_swap(a1, b1, false, false);
                    u32x4 o; o.x = r0[0]; o.y = r1[0]; o.z = r0[1]; o.w = r1[1];
                    if (bad) o = (u32x4){0x7fc07fc0u, 0x7fc07fc0u, 0x7fc07fc0u, 0x7fc07fc0u};
                    *(u32x4*)(rowp + bj * 256) = o;
                }
            }
    }
    if (tid == 0) __hip_atomic_fetch_add((GAS unsigned*)(myws + WS_CNT), 1u, __ATOMIC_RELAXED, __HIP_MEMORY_SCOPE_AGENT);
}

extern "C" void kernel_launch(int rank, int nranks, ncclComm_t comm,
                              hipStream_t stream,
                              void* const* d_in, const int* in_sizes, int n_in,
                              void* d_out, int out_size,
                              const int* in_dtypes, int out_dtype,
                              void* const* d_out_all,
                              void* const* d_ws_all, size_t ws_bytes) {
    (void)comm; (void)in_dtypes; (void)out_dtype; (void)d_out_all;
    if (nranks != NR || n_in < 2 || rank < 0 || rank >= NR || ws_bytes < WS_NEED ||
        in_sizes[0] != KDIM * KC || in_sizes[1] != KDIM * NDIM || out_size != MLOC * NDIM) {
        fprintf(stderr, "kernel_launch: unexpected problem shape\n");
        return;
    }
    static bool first = true;
    if (first) {
        HIP_CHECK(hipFuncSetAttribute((const void*)a2a_gemm_kernel, hipFuncAttributeMaxDynamicSharedMemorySize, LDS_BYTES));
        first = false;
    }
    Params p;
    p.x = (const char*)d_in[0];
    p.w = (const char*)d_in[1];
    p.out = (char*)d_out;
    for (int i = 0; i < NR; ++i) p.ws[i] = (char*)d_ws_all[i];
    p.rank = rank;
    a2a_gemm_kernel<<<GRID, 512, LDS_BYTES, stream>>>(p);
    HIP_CHECK(hipGetLastError());
}
```

```cpp
#include <hip/hip_runtime.h>
#include <rccl/rccl.h>
#include <cstdio>
#include <cstdint>

#define HIP_CHECK(expr) do { hipError_t _e = (expr); if (_e != hipSuccess) { \
    fprintf(stderr, "HIP error %s at %s:%d\n", hipGetErrorString(_e), __FILE__, __LINE__); \
    return; } } while (0)

#define LAS __attribute__((address_space(3)))
#define GAS __attribute__((address_space(1)))
typedef short s16x8 __attribute__((ext_vector_type(8)));
typedef short s16x4 __attribute__((ext_vector_type(4)));
typedef short s16x2 __attribute__((ext_vector_type(2)));
typedef __bf16 bf16x8 __attribute__((ext_vector_type(8)));
typedef float f32x4 __attribute__((ext_vector_type(4)));
typedef unsigned u32x4 __attribute__((ext_vector_type(4)));

constexpr int NR = 8;
constexpr int MLOC = 512;
constexpr int KDIM = 4096;
constexpr int NDIM = 32768;
constexpr int KC = KDIM / NR;
constexpr unsigned CHUNK_BYTES = MLOC * KC * 2;
constexpr int HTB = 16384;
constexpr int LDS_BYTES = 8 * HTB;
constexpr unsigned W_BYTES = (unsigned)KDIM * NDIM * 2u;
constexpr unsigned WROW = NDIM * 2;
constexpr unsigned AROW = KC * 2;
constexpr int GRID = 256;
constexpr size_t WS_CNT = 0;
constexpr size_t WS_TMO = 128;
constexpr size_t WS_FLAGS = 4096;
constexpr size_t WS_STAGE = 1u << 20;
constexpr size_t SLOT_BYTES = (size_t)NR * CHUNK_BYTES;
constexpr size_t WS_NEED = WS_STAGE + 2 * SLOT_BYTES;
constexpr unsigned long long DEADLINE_TICKS = 40000000ull;

#define STAMP(i) do { } while (0)
#define STAMPX_BEGIN do { } while (0)
#define STAMPX_END(g) do { } while (0)
#define OUT_ST "sc1 nt"
#define COMM_ON true
struct Params {
    const char* x;
    const char* w;
    char* out;
    char* ws[NR];
    int rank;
};

__device__ __forceinline__ unsigned cvt_pk_bf16(float lo, float hi) {
    unsigned r; asm("v_cvt_pk_bf16_f32 %0, %1, %2" : "=v"(r) : "v"(lo), "v"(hi)); return r;
}
__device__ __forceinline__ unsigned relu_pk(unsigned v) {
    s16x2 a = __builtin_bit_cast(s16x2, v);
    s16x2 z = {0, 0};
    return __builtin_bit_cast(unsigned, __builtin_elementwise_max(a, z));
}

template <class T> __device__ __forceinline__ T* uni(T* ptr) {
    const unsigned long long v = (unsigned long long)ptr;
    const unsigned lo = __builtin_amdgcn_readfirstlane((unsigned)v), hi = __builtin_amdgcn_readfirstlane((unsigned)(v >> 32));
    return (T*)(((unsigned long long)hi << 32) | lo);
}

__device__ __forceinline__ void pair_map(int u, int rank, int& chunk, int& sub) {
    if (u < 4) { chunk = rank; sub = u; }
    else { const int v = u - 4; const int g = (v * 37) >> 8;   chunk = (rank + 1 + (v - g * 7)) & 7; sub = g; }
}

__global__ __launch_bounds__(512, 2) void a2a_gemm_kernel(Params p) {
    extern __shared__ __attribute__((aligned(16))) char smem_[];
    LAS char* lds = (LAS char*)smem_;
    const int tid = threadIdx.x;
    const int wid = __builtin_amdgcn_readfirstlane(tid >> 6);
    const int lane = tid & 63, wr = wid >> 2, wc = wid & 3, fr = lane & 15, fq = lane >> 4;
    const int bid = blockIdx.x;
    const int xcd = bid & 7, jj = bid >> 3;
    const int pm = jj & 1, pn = xcd * 16 + (jj >> 1);
    const int rank = p.rank;
    char* const myws = uni(p.ws[rank]);
    STAMP(0);

    unsigned cntv;
    const unsigned vzero = 0;
    asm volatile("global_load_dword %0, %1, %2 sc1" : "=v"(cntv) : "v"(vzero), "s"(myws + WS_CNT) : "memory");

    const bool pusher = COMM_ON && (wid == 1) && (bid < 224);
    const unsigned dest = (unsigned)(rank + 1 + (bid >> 5)) & 7u;
    const unsigned pj = (unsigned)bid & 31u;
    const unsigned voffP = (unsigned)(lane >> 4) * AROW + (unsigned)(lane & 15) * 16u;
    const char* const psrc = uni(p.x + (size_t)(dest * MLOC + pj * 16u) * AROW);
    char* const dws = uni(p.ws[dest]);
    u32x4 pd0, pd1, pd2, pd3;
    if (pusher) {
        asm volatile("global_load_dwordx4 %0, %1, %2" : "=v"(pd0) : "v"(voffP), "s"(psrc) : "memory");
        asm volatile("global_load_dwordx4 %0, %1, %2" : "=v"(pd1) : "v"(voffP), "s"(psrc + 4 * AROW) : "memory");
        asm volatile("global_load_dwordx4 %0, %1, %2" : "=v"(pd2) : "v"(voffP), "s"(psrc + 8 * AROW) : "memory");
        asm volatile("global_load_dwordx4 %0, %1, %2" : "=v"(pd3) : "v"(voffP), "s"(psrc + 12 * AROW) : "memory");
    }

    const unsigned voffA = (unsigned)(lane >> 3) * AROW + (unsigned)(((lane & 7) ^ (lane >> 3)) * 16);
    const unsigned voffB = (unsigned)(((lane >> 3) & 1) * 8 + (lane >> 4)) * WROW + (unsigned)(((lane & 7) ^ ((lane >> 4) << 1)) * 16);
    const unsigned sA_w = (unsigned)(pm * 256 + wid * 8) * AROW;
    const unsigned sB_w = (unsigned)(((wid >> 2) & 1) * 16 + ((wid >> 1) & 1) * 4) * WROW + (unsigned)(wid & 1) * 128u + (unsigned)pn * 512u;
    const unsigned ldsw = (unsigned)wid * 1024u;
    const int arow = wr * 64 + fr;
    const unsigned aoff0 = (unsigned)arow * 128u + (unsigned)((fq ^ (fr & 7)) * 16);
    const unsigned aoff1 = aoff0 ^ 64u;
    const int q = (lane & 15) >> 2, pp = lane & 3;
    const unsigned bcom = (unsigned)(fq >> 1) * 4096u + (unsigned)(wc >> 1) * 1024u + (unsigned)q * 256u + (unsigned)(fq & 1) * 128u + (unsigned)pp * 8u;
    const unsigned ldsbase = (unsigned)(size_t)lds;
    const unsigned boff0 = ldsbase + 4 * HTB + bcom + (unsigned)(((2 * (wc & 1) + 0) ^ q) * 32);
    const unsigned boff1 = ldsbase + 4 * HTB + bcom + (unsigned)(((2 * (wc & 1) + 1) ^ q) * 32);

#define SA(b, h) (((b) * 2 + (h)) * HTB)
#define SB(b, h) (((b) * 2 + (h)) * HTB)
#define STAGE_A(b, h, rs, so) do { \
        __builtin_amdgcn_raw_ptr_buffer_load_lds(rs, (LAS void*)(lds + SA(b, h) + ldsw), 16, voffA, (so) + sA_w + (h) * 128u * AROW, 0, 0); \
        __builtin_amdgcn_raw_ptr_buffer_load_lds(rs, (LAS void*)(lds + SA(b, h) + ldsw + 8192), 16, voffA, (so) + sA_w + (h) * 128u * AROW + 64u * AROW, 0, 0); } while (0)
#define STAGE_B(b, h, rs, so) do { \
        __builtin_amdgcn_raw_ptr_buffer_load_lds(rs, (LAS void*)(lds + 4 * HTB + SB(b, h) + ldsw), 16, voffB, (so) + sB_w + (h) * 256u, 0, 0); \
        __builtin_amdgcn_raw_ptr_buffer_load_lds(rs, (LAS void*)(lds + 4 * HTB + SB(b, h) + ldsw + 8192), 16, voffB, (so) + sB_w + (h) * 256u + 32u * WROW, 0, 0); } while (0)
#define LDA(dst, b, h) do { _Pragma("unroll") for (int m = 0; m < 4; ++m) { \
        dst[m][0] = *(const LAS s16x8*)(lds + SA(b, h) + aoff0 + m * 2048); \
        dst[m][1] = *(const LAS s16x8*)(lds + SA(b, h) + aoff1 + m * 2048); } } while (0)
#define TRR(dst, base, off) asm volatile("ds_read_b64_tr_b16 %0, %1 offset:%2" : "=v"(dst) : "v"(base), "i"(off) : "memory")
#define LDB(dst, b, h) do { _Pragma("unroll") for (int k = 0; k < 2; ++k) { \
        { s16x4 lo, hi; TRR(lo, boff0, SB(b, h) + k * 8192); TRR(hi, boff0, SB(b, h) + k * 8192 + 2048); dst[0][k] = __builtin_shufflevector(lo, hi, 0, 1, 2, 3, 4, 5, 6, 7); } \
        { s16x4 lo, hi; TRR(lo, boff1, SB(b, h) + k * 8192); TRR(hi, boff1, SB(b, h) + k * 8192 + 2048); dst[1][k] = __builtin_shufflevector(lo, hi, 0, 1, 2, 3, 4, 5, 6, 7); } } } while (0)
#define MMA(ai, bj, At, Bt) do { __builtin_amdgcn_s_setprio(1); _Pragma("unroll") for (int m = 0; m < 4; ++m) _Pragma("unroll") for (int n = 0; n < 2; ++n) _Pragma("unroll") for (int k = 0; k < 2; ++k) \
        acc[ai][bj][m][n] = __builtin_amdgcn_mfma_f32_16x16x32_bf16(__builtin_bit_cast(bf16x8, Bt[n][k]), __builtin_bit_cast(bf16x8, At[m][k]), acc[ai][bj][m][n], 0, 0, 0); __builtin_amdgcn_s_setprio(0); } while (0)
#define WAIT_V(n) asm volatile("s_waitcnt vmcnt(" #n ")" ::: "memory")
#define WAIT_L(n) asm volatile("s_waitcnt lgkmcnt(" #n ")" ::: "memory")
#define BAR __builtin_amdgcn_s_barrier()
#define SCHED __builtin_amdgcn_sched_barrier(0)
#define PSTORE(data, base) asm volatile("global_store_dwordx4 %0, %1, %2 sc0 sc1\n\ts_nop 1" :: "v"(voffP), "v"(data), "s"(base) : "memory")

    f32x4 acc[2][2][4][2];
#pragma unroll
    for (int a = 0; a < 2; ++a)
#pragma unroll
        for (int b = 0; b < 2; ++b)
#pragma unroll
            for (int m = 0; m < 4; ++m)
#pragma unroll
                for (int n = 0; n < 2; ++n) acc[a][b][m][n] = (f32x4){0.f, 0.f, 0.f, 0.f};
    s16x8 At[4][2], B0[2][2], B1[2][2];

    const auto rW = __builtin_amdgcn_make_buffer_rsrc((void*)p.w, 0, W_BYTES, 0x00020000);
    const auto rWz = __builtin_amdgcn_make_buffer_rsrc((void*)p.w, 0, 0, 0x00020000);
    const char* const xown = uni(p.x + (size_t)rank * CHUNK_BYTES);

    auto rA = __builtin_amdgcn_make_buffer_rsrc((void*)xown, 0, CHUNK_BYTES, 0x00020000);
    unsigned soA = 0;
    unsigned soB = (unsigned)(rank * KC) * WROW;

    STAGE_B(0, 0, rW, soB); STAGE_A(0, 0, rA, soA); STAGE_B(0, 1, rW, soB); STAGE_A(0, 1, rA, soA);
    STAGE_B(1, 0, rW, soB + 64u * WROW); STAGE_A(1, 0, rA, soA + 128u); STAGE_B(1, 1, rW, soB + 64u * WROW);
    WAIT_V(14);
    asm volatile("" : "+v"(cntv));
    const unsigned call = (unsigned)__builtin_amdgcn_readfirstlane(cntv) / GRID + 1u;
    const size_t slot_off = WS_STAGE + (size_t)(call & 1u) * SLOT_BYTES;
    const char* const stage = uni(myws + slot_off);
    char* const pdst = uni(dws + slot_off + (size_t)rank * CHUNK_BYTES + (size_t)(pj * 16u) * AROW);
    char* const pflag = uni(dws + WS_FLAGS + (size_t)rank * 128 + pj * 4u);
    STAMP(1);
    if (pusher) {
        asm volatile("" : "+v"(pd0), "+v"(pd1), "+v"(pd2), "+v"(pd3));
        PSTORE(pd0, pdst); PSTORE(pd1, pdst + 4 * AROW); PSTORE(pd2, pdst + 8 * AROW); PSTORE(pd3, pdst + 12 * AROW);
        WAIT_V(14);
    } else {
        WAIT_V(10);
    }
    BAR;
    LDB(B0, 0, 0);
    if (wr == 1) BAR;
    STAMP(2);

    auto chunk_base = [&](int c) -> const char* { return c == rank ? xown : stage + (size_t)c * CHUNK_BYTES; };
    unsigned tmo_val = 0;
    u32x4 fl;
    const unsigned vcall = call;
#define PFLAG(g) do { if (lane == 0) asm volatile("global_store_dword %0, %1, %2 sc0 sc1" :: "v"(vzero), "v"(vcall), "s"(pflag + (g) * 1024) : "memory"); } while (0)
#define FLAG_VOFF ((unsigned)(((rank + 1 + (lane >> 3)) & 7) * 128 + (lane & 7) * 16))
#define PH(LOAD, STAGE, HOOK, MM) do { LOAD; STAGE; WAIT_V(10); HOOK; BAR; WAIT_L(0); SCHED; MM; BAR; SCHED; } while (0)
#define PH4(LOAD, STAGE, HOOK, MM) do { LOAD; STAGE; WAIT_V(10); HOOK; BAR; SCHED; MM; BAR; SCHED; } while (0)
    for (int u = 0; u < 32; ++u) {
        const bool last = (u == 31);
        STAMP(3 + u);
        int ch2, sub2;
        pair_map(last ? 31 : u + 1, rank, ch2, sub2);
        const auto rA2 = __builtin_amdgcn_make_buffer_rsrc((void*)chunk_base(ch2), 0, last ? 0 : CHUNK_BYTES, 0x00020000);
        const auto rW2 = last ? rWz : rW;
        const unsigned soA2 = (unsigned)sub2 * 256u;
        const unsigned soB2 = (unsigned)(ch2 * KC + sub2 * 128) * WROW;
        const bool pact = pusher && u < 12;
        const unsigned poff = (unsigned)(u & 3) * 4u * AROW + (unsigned)(1 + (u >> 2)) * 256u;
        const bool pflg = pusher && (u & 3) == 0 && u <= 12;
        const int cg = (u * 37) >> 8;
        const int cr = u - cg * 7;
        const bool cload = COMM_ON && (wid == 0) && cr == 1 && cg < 4;
        const bool ceval = COMM_ON && (wid == 0) && cr == 2 && cg < 4;
#define HK_PLOAD do { if (pact) asm volatile("global_load_dwordx4 %0, %1, %2" : "=v"(pd0) : "v"(voffP), "s"(psrc + poff) : "memory"); } while (0)
#define HK_PSTORE do { if (pact) { asm volatile("" : "+v"(pd0)); PSTORE(pd0, pdst + poff); } \
                       if (cload) asm volatile("global_load_dwordx4 %0, %1, %2 sc0 sc1" : "=v"(fl) : "v"(FLAG_VOFF), "s"(myws + WS_FLAGS + cg * 1024) : "memory"); \
                       if (last) asm volatile("global_load_dword %0, %1, %2 sc1" : "=v"(tmo_val) : "v"(vzero), "s"(myws + WS_TMO) : "memory"); } while (0)
#define HK_PFLAG do { if (pflg) PFLAG(u >> 2); } while (0)
#define HK_CEVAL do { if (ceval) { \
            asm volatile("" : "+v"(fl)); \
            bool ok = lane >= 56 || (fl.x >= call && fl.y >= call && fl.z >= call && fl.w >= call); \
            if (!__all(ok)) { \
                STAMPX_BEGIN; \
                const unsigned long long t0 = __builtin_amdgcn_s_memrealtime(); \
                for (;;) { \
                    unsigned tm; \
                    asm volatile("global_load_dwordx4 %0, %2, %3 sc0 sc1\n\tglobal_load_dword %1, %4, %5 sc1\n\ts_waitcnt vmcnt(0)" : "=&v"(fl), "=&v"(tm) \
                                 : "v"(FLAG_VOFF), "s"(myws + WS_FLAGS + cg * 1024), "v"(vzero), "s"(myws + WS_TMO) : "memory"); \
                    ok = lane >= 56 || (fl.x >= call && fl.y >= call && fl.z >= call && fl.w >= call); \
                    if (__all(ok)) break; \
                    if ((unsigned)__builtin_amdgcn_readfirstlane(tm) == call || __builtin_amdgcn_s_memrealtime() - t0 > DEADLINE_TICKS) { \
                        if (lane == 0) __hip_atomic_store((GAS unsigned*)(myws + WS_TMO), call, __ATOMIC_RELAXED, __HIP_MEMORY_SCOPE_AGENT); \
                        asm volatile("s_waitcnt vmcnt(0)" ::: "memory"); \
                        break; } \
                    __builtin_amdgcn_s_sleep(2); \
                } \
                STAMPX_END(cg); \
            } \
            asm volatile("buffer_inv sc0 sc1" ::: "memory"); } } while (0)
#define HK_NONE do { } while (0)
        PH(LDA(At, 0, 0), STAGE_A(1, 1, rA, soA + 128u), HK_PLOAD, MMA(0, 0, At, B0));
        PH(LDB(B1, 0, 1), STAGE_B(0, 0, rW2, soB2), HK_NONE, MMA(0, 1, At, B1));
        PH(LDA(At, 0, 1), STAGE_A(0, 0, rA2, soA2), HK_NONE, MMA(1, 0, At, B0));
        PH4(LDB(B0, 1, 0), STAGE_B(0, 1, rW2, soB2), HK_NONE, MMA(1, 1, At, B1));
        PH(LDA(At, 1, 0), STAGE_A(0, 1, rA2, soA2), HK_CEVAL, MMA(0, 0, At, B0));
        PH(LDB(B1, 1, 1), STAGE_B(1, 0, rW2, soB2 + 64u * WROW), HK_PFLAG, MMA(0, 1, At, B1));
        PH(LDA(At, 1, 1), STAGE_A(1, 0, rA2, soA2 + 128u), HK_PSTORE, MMA(1, 0, At, B0));
        PH4(LDB(B0, 0, 0), STAGE_B(1, 1, rW2, soB2 + 64u * WROW), HK_NONE, MMA(1, 1, At, B1));
        rA = rA2; soA = soA2; soB = soB2;
    }
    STAMP(35);
    WAIT_V(0); WAIT_L(0);
    if (wr == 0) BAR;

    {
        asm volatile("" : "+v"(tmo_val));
        int tz = threadIdx.x; asm volatile("" : "+v"(tz));
        const int l2 = tz & 63, fr2 = l2 & 15, fq2 = l2 >> 4;
        const bool bad = (tmo_val == call);
        char* obase = p.out + (size_t)(pm * 256 + wr * 64 + fr2) * WROW + (size_t)pn * 512 + wc * 64 + ((fq2 & 1) * 32 + (fq2 >> 1) * 16);
#pragma unroll
        for (int ai = 0; ai < 2; ++ai)
#pragma unroll
            for (int m = 0; m < 4; ++m) {
                char* rowp = obase + (size_t)(ai * 128 + m * 16) * WROW;
#pragma unroll
                for (int bj = 0; bj < 2; ++bj) {
                    const f32x4 v0 = acc[ai][bj][m][0], v1 = acc[ai][bj][m][1];
                    unsigned a0 = relu_pk(cvt_pk_bf16(v0[0], v0[1])), a1 = relu_pk(cvt_pk_bf16(v0[2], v0[3]));
                    unsigned b0 = relu_pk(cvt_pk_bf16(v1[0], v1[1])), b1 = relu_pk(cvt_pk_bf16(v1[2], v1[3]));
                    auto r0 = __builtin_amdgcn_permlane16_swap(a0, b0, false, false);
                    auto r1 = __builtin_amdgcn_permlane16_swap(a1, b1, false, false);
                    u32x4 o; o.x = r0[0]; o.y = r1[0]; o.z = r0[1]; o.w = r1[1];
                    if (bad) o = (u32x4){0x7fc07fc0u, 0x7fc07fc0u, 0x7fc07fc0u, 0x7fc07fc0u};
                    asm volatile("global_store_dwordx4 %0, %1, off " OUT_ST "\n\ts_nop 1" :: "v"(rowp + bj * 256), "v"(o) : "memory");
                }
            }
    }
    if (tid == 0) __hip_atomic_fetch_add((GAS unsigned*)(myws + WS_CNT), 1u, __ATOMIC_RELAXED, __HIP_MEMORY_SCOPE_AGENT);
}

extern "C" void kernel_launch(int rank, int nranks, ncclComm_t comm,
                              hipStream_t stream,
                              void* const* d_in, const int* in_sizes, int n_in,
                              void* d_out, int out_size,
                              const int* in_dtypes, int out_dtype,
                              void* const* d_out_all,
                              void* const* d_ws_all, size_t ws_bytes) {
    (void)comm; (void)in_dtypes; (void)out_dtype; (void)d_out_all;
    if (nranks != NR || n_in < 2 || rank < 0 || rank >= NR || ws_bytes < WS_NEED ||
        in_sizes[0] != KDIM * KC || in_sizes[1] != KDIM * NDIM || out_size != MLOC * NDIM) {
        fprintf(stderr, "kernel_launch: unexpected problem shape\n");
        return;
    }
    static bool first = true;
    if (first) {
        HIP_CHECK(hipFuncSetAttribute((const void*)a2a_gemm_kernel, hipFuncAttributeMaxDynamicSharedMemorySize, LDS_BYTES));
        first = false;
    }
    Params p;
    p.x = (const char*)d_in[0];
    p.w = (const char*)d_in[1];
    p.out = (char*)d_out;
    for (int i = 0; i < NR; ++i) p.ws[i] = (char*)d_ws_all[i];
    p.rank = rank;
    a2a_gemm_kernel<<<GRID, 512, LDS_BYTES, stream>>>(p);
    HIP_CHECK(hipGetLastError());
}
```

```cpp
#include <hip/hip_runtime.h>
#include <rccl/rccl.h>
#include <cstdio>
#include <cstdint>

#define HIP_CHECK(expr) do { hipError_t _e = (expr); if (_e != hipSuccess) { \
    fprintf(stderr, "HIP error %s at %s:%d\n", hipGetErrorString(_e), __FILE__, __LINE__); \
    return; } } while (0)

#define LAS __attribute__((address_space(3)))
#define GAS __attribute__((address_space(1)))
typedef short s16x8 __attribute__((ext_vector_type(8)));
typedef short s16x4 __attribute__((ext_vector_type(4)));
typedef short s16x2 __attribute__((ext_vector_type(2)));
typedef __bf16 bf16x8 __attribute__((ext_vector_type(8)));
typedef float f32x4 __attribute__((ext_vector_type(4)));
typedef unsigned u32x4 __attribute__((ext_vector_type(4)));

constexpr int NR = 8;
constexpr int MLOC = 512;
constexpr int KDIM = 4096;
constexpr int NDIM = 32768;
constexpr int KC = KDIM / NR;
constexpr unsigned CHUNK_BYTES = MLOC * KC * 2;
constexpr int HTB = 16384;
constexpr int LDS_BYTES = 8 * HTB;
constexpr unsigned W_BYTES = (unsigned)KDIM * NDIM * 2u;
constexpr unsigned WROW = NDIM * 2;
constexpr unsigned AROW = KC * 2;
constexpr int GRID = 256;
constexpr size_t WS_CNT = 0;
constexpr size_t WS_TMO = 128;
constexpr size_t WS_FIN = 256;
constexpr size_t WS_FLAGS = 4096;
constexpr size_t WS_STAGE = 1u << 20;
constexpr size_t SLOT_BYTES = (size_t)NR * CHUNK_BYTES;
constexpr size_t WS_NEED = WS_STAGE + 2 * SLOT_BYTES;
constexpr unsigned long long DEADLINE_TICKS = 40000000ull;

#define STAMP(i) do { } while (0)
#define STAMPC(i) do { } while (0)
#define STAMPX_BEGIN do { } while (0)
#define STAMPX_END(g) do { } while (0)
#define W_AUX 0
#define OUT_ST "sc1 nt"
#define COMM_ON true
#define THR_N 0
#define THR_SLEEP 8
#define INV_MODE 4
struct Params {
    const char* x;
    const char* w;
    char* out;
    char* ws[NR];
    int rank;
};

__device__ __forceinline__ unsigned cvt_pk_bf16(float lo, float hi) {
    unsigned r; asm("v_cvt_pk_bf16_f32 %0, %1, %2" : "=v"(r) : "v"(lo), "v"(hi)); return r;
}
__device__ __forceinline__ unsigned relu_pk(unsigned v) {
    s16x2 a = __builtin_bit_cast(s16x2, v);
    s16x2 z = {0, 0};
    return __builtin_bit_cast(unsigned, __builtin_elementwise_max(a, z));
}

template <class T> __device__ __forceinline__ T* uni(T* ptr) {
    const unsigned long long v = (unsigned long long)ptr;
    const unsigned lo = __builtin_amdgcn_readfirstlane((unsigned)v), hi = __builtin_amdgcn_readfirstlane((unsigned)(v >> 32));
    return (T*)(((unsigned long long)hi << 32) | lo);
}

__device__ __forceinline__ void pair_map(int u, int rank, int& chunk, int& sub) {
    if (u < 4) { chunk = rank; sub = u; }
    else { const int v = u - 4; const int g = (v * 37) >> 8;   chunk = (rank + 1 + (v - g * 7)) & 7; sub = g; }
}

__global__ __launch_bounds__(512, 2) void a2a_gemm_kernel(Params p) {
    extern __shared__ __attribute__((aligned(16))) char smem_[];
    LAS char* lds = (LAS char*)smem_;
    const int tid = threadIdx.x;
    const int wid = __builtin_amdgcn_readfirstlane(tid >> 6);
    const int lane = tid & 63, wr = wid >> 2, wc = wid & 3, fr = lane & 15, fq = lane >> 4;
    const int bid = blockIdx.x;
    const int xcd = bid & 7, jj = bid >> 3;
    const int pm = jj & 1, pn = xcd * 16 + (jj >> 1);
    const int rank = p.rank;
    char* const myws = uni(p.ws[rank]);
    STAMP(0);

    const unsigned vzero = 0;
    asm volatile("global_load_dword v242, %0, %1 sc1" :: "v"(vzero), "s"(myws + WS_CNT) : "memory", "v242");

    if (INV_MODE == 1 && wid == 0) asm volatile("buffer_inv sc0 sc1" ::: "memory");

    const bool pusher = COMM_ON && (wid == 1) && (bid < 224);
    const unsigned dest = (unsigned)(rank + 1 + (bid >> 5)) & 7u;
    const unsigned pj = (unsigned)bid & 31u;
    const unsigned voffP = (unsigned)(lane >> 4) * AROW + (unsigned)(lane & 15) * 16u;
    const char* const psrc = uni(p.x + (size_t)(dest * MLOC + pj * 16u) * AROW);
    char* const dws = uni(p.ws[dest]);
    if (pusher) {
        asm volatile("global_load_dwordx4 v[232:235], %0, %1" :: "v"(voffP), "s"(psrc) : "memory", "v232", "v233", "v234", "v235");
        asm volatile("global_load_dwordx4 v[244:247], %0, %1" :: "v"(voffP), "s"(psrc + 4 * AROW) : "memory", "v244", "v245", "v246", "v247");
        asm volatile("global_load_dwordx4 v[248:251], %0, %1" :: "v"(voffP), "s"(psrc + 8 * AROW) : "memory", "v248", "v249", "v250", "v251");
        asm volatile("global_load_dwordx4 v[252:255], %0, %1" :: "v"(voffP), "s"(psrc + 12 * AROW) : "memory", "v252", "v253", "v254", "v255");
    }

    const unsigned voffA = (unsigned)(lane >> 3) * AROW + (unsigned)(((lane & 7) ^ (lane >> 3)) * 16);
    const unsigned voffB = (unsigned)(((lane >> 3) & 1) * 8 + (lane >> 4)) * WROW + (unsigned)(((lane & 7) ^ ((lane >> 4) << 1)) * 16);
    const unsigned sA_w = (unsigned)(pm * 256 + wid * 8) * AROW;
    const unsigned sB_w = (unsigned)(((wid >> 2) & 1) * 16 + ((wid >> 1) & 1) * 4) * WROW + (unsigned)(wid & 1) * 128u + (unsigned)pn * 512u;
    const unsigned ldsw = (unsigned)wid * 1024u;
    const int arow = wr * 64 + fr;
    const unsigned aoff0 = (unsigned)arow * 128u + (unsigned)((fq ^ (fr & 7)) * 16);
    const unsigned aoff1 = aoff0 ^ 64u;
    const int q = (lane & 15) >> 2, pp = lane & 3;
    const unsigned bcom = (unsigned)(fq >> 1) * 4096u + (unsigned)(wc >> 1) * 1024u + (unsigned)q * 256u + (unsigned)(fq & 1) * 128u + (unsigned)pp * 8u;
    const unsigned ldsbase = (unsigned)(size_t)lds;
    const unsigned boff0 = ldsbase + 4 * HTB + bcom + (unsigned)(((2 * (wc & 1) + 0) ^ q) * 32);
    const unsigned boff1 = ldsbase + 4 * HTB + bcom + (unsigned)(((2 * (wc & 1) + 1) ^ q) * 32);

#define SA(b, h) (((b) * 2 + (h)) * HTB)
#define SB(b, h) (((b) * 2 + (h)) * HTB)
#define STAGE_A(b, h, rs, so) do { \
        __builtin_amdgcn_raw_ptr_buffer_load_lds(rs, (LAS void*)(lds + SA(b, h) + ldsw), 16, voffA, (so) + sA_w + (h) * 128u * AROW, 0, 0); \
        __builtin_amdgcn_raw_ptr_buffer_load_lds(rs, (LAS void*)(lds + SA(b, h) + ldsw + 8192), 16, voffA, (so) + sA_w + (h) * 128u * AROW + 64u * AROW, 0, 0); } while (0)
#define STAGE_B(b, h, rs, so) do { \
        __builtin_amdgcn_raw_ptr_buffer_load_lds(rs, (LAS void*)(lds + 4 * HTB + SB(b, h) + ldsw), 16, voffB, (so) + sB_w + (h) * 256u, 0, W_AUX); \
        __builtin_amdgcn_raw_ptr_buffer_load_lds(rs, (LAS void*)(lds + 4 * HTB + SB(b, h) + ldsw + 8192), 16, voffB, (so) + sB_w + (h) * 256u + 32u * WROW, 0, W_AUX); } while (0)
#define LDA(dst, b, h) do { _Pragma("unroll") for (int m = 0; m < 4; ++m) { \
        dst[m][0] = *(const LAS s16x8*)(lds + SA(b, h) + aoff0 + m * 2048); \
        dst[m][1] = *(const LAS s16x8*)(lds + SA(b, h) + aoff1 + m * 2048); } } while (0)
#define TRR(dst, base, off) asm volatile("ds_read_b64_tr_b16 %0, %1 offset:%2" : "=v"(dst) : "v"(base), "i"(off) : "memory")
#define LDB(dst, b, h) do { _Pragma("unroll") for (int k = 0; k < 2; ++k) { \
        { s16x4 lo, hi; TRR(lo, boff0, SB(b, h) + k * 8192); TRR(hi, boff0, SB(b, h) + k * 8192 + 2048); dst[0][k] = __builtin_shufflevector(lo, hi, 0, 1, 2, 3, 4, 5, 6, 7); } \
        { s16x4 lo, hi; TRR(lo, boff1, SB(b, h) + k * 8192); TRR(hi, boff1, SB(b, h) + k * 8192 + 2048); dst[1][k] = __builtin_shufflevector(lo, hi, 0, 1, 2, 3, 4, 5, 6, 7); } } } while (0)
#define MMA(ai, bj, At, Bt) do { __builtin_amdgcn_s_setprio(1); _Pragma("unroll") for (int m = 0; m < 4; ++m) _Pragma("unroll") for (int n = 0; n < 2; ++n) _Pragma("unroll") for (int k = 0; k < 2; ++k) \
        acc[ai][bj][m][n] = __builtin_amdgcn_mfma_f32_16x16x32_bf16(__builtin_bit_cast(bf16x8, Bt[n][k]), __builtin_bit_cast(bf16x8, At[m][k]), acc[ai][bj][m][n], 0, 0, 0); __builtin_amdgcn_s_setprio(0); } while (0)
#define MMAK(ai, bj, At, Bt, k) do { _Pragma("unroll") for (int m = 0; m < 4; ++m) _Pragma("unroll") for (int n = 0; n < 2; ++n) \
        acc[ai][bj][m][n] = __builtin_amdgcn_mfma_f32_16x16x32_bf16(__builtin_bit_cast(bf16x8, Bt[n][k]), __builtin_bit_cast(bf16x8, At[m][k]), acc[ai][bj][m][n], 0, 0, 0); } while (0)
#define MMAH(ai, bj, At, Bt, HOOK) do { __builtin_amdgcn_s_setprio(1); MMAK(ai, bj, At, Bt, 0); SCHED; HOOK; SCHED; MMAK(ai, bj, At, Bt, 1); __builtin_amdgcn_s_setprio(0); } while (0)
#define WAIT_V(n) asm volatile("s_waitcnt vmcnt(" #n ")" ::: "memory")
#define WAIT_L(n) asm volatile("s_waitcnt lgkmcnt(" #n ")" ::: "memory")
#define BAR __builtin_amdgcn_s_barrier()
#define SCHED __builtin_amdgcn_sched_barrier(0)
#define PSTORE(areg, base) asm volatile("global_store_dwordx4 %0, " areg ", %1 sc0 sc1\n\ts_nop 1" :: "v"(voffP), "s"(base) : "memory")

    f32x4 acc[2][2][4][2];
#pragma unroll
    for (int a = 0; a < 2; ++a)
#pragma unroll
        for (int b = 0; b < 2; ++b)
#pragma unroll
            for (int m = 0; m < 4; ++m)
#pragma unroll
                for (int n = 0; n < 2; ++n) acc[a][b][m][n] = (f32x4){0.f, 0.f, 0.f, 0.f};
    s16x8 At[4][2], B0[2][2], B1[2][2];

    const auto rW = __builtin_amdgcn_make_buffer_rsrc((void*)p.w, 0, W_BYTES, 0x00020000);
    const auto rWz = __builtin_amdgcn_make_buffer_rsrc((void*)p.w, 0, 0, 0x00020000);
    const char* const xown = uni(p.x + (size_t)rank * CHUNK_BYTES);

    auto rA = __builtin_amdgcn_make_buffer_rsrc((void*)xown, 0, CHUNK_BYTES, 0x00020000);
    unsigned soA = 0;
    unsigned soB = (unsigned)(rank * KC) * WROW;

    STAGE_B(0, 0, rW, soB); STAGE_A(0, 0, rA, soA); STAGE_B(0, 1, rW, soB); STAGE_A(0, 1, rA, soA);
    WAIT_V(8);
    unsigned cntv;
    asm volatile("v_mov_b32 %0, v242" : "=v"(cntv) :: "memory");
    const unsigned call = (unsigned)__builtin_amdgcn_readfirstlane(cntv) / GRID + 1u;
    const size_t slot_off = WS_STAGE + (size_t)(call & 1u) * SLOT_BYTES;
    const char* const stage = uni(myws + slot_off);
    char* const pdst = uni(dws + slot_off + (size_t)rank * CHUNK_BYTES + (size_t)(pj * 16u) * AROW);
    char* const pflag = uni(dws + WS_FLAGS + (size_t)rank * 128 + pj * 4u);
    STAMP(1);
    if (pusher) {
        PSTORE("v[232:235]", pdst); PSTORE("v[244:247]", pdst + 4 * AROW); PSTORE("v[248:251]", pdst + 8 * AROW); PSTORE("v[252:255]", pdst + 12 * AROW);
        STAMP(38);
        WAIT_V(8);
    } else {
        WAIT_V(4);
    }
    STAGE_B(1, 0, rW, soB + 64u * WROW); STAGE_A(1, 0, rA, soA + 128u); STAGE_B(1, 1, rW, soB + 64u * WROW);
    STAMP(39);
    BAR;
    LDB(B0, 0, 0);
    if (wr == 1) BAR;
    STAMP(2);

    const unsigned vcall = call;
#define PFLAG(g) do { if (lane == 0) asm volatile("global_store_dword %0, %1, %2 sc0 sc1" :: "v"(vzero), "v"(vcall), "s"(pflag + (g) * 1024) : "memory"); } while (0)
#define FLAG_VOFF ((unsigned)(((rank + 1 + (lane >> 3)) & 7) * 128 + (lane & 7) * 16))
#define PH(LOAD, STAGE, MM) do { LOAD; STAGE; WAIT_V(10); BAR; WAIT_L(0); SCHED; MM; BAR; SCHED; } while (0)
#define PH4(LOAD, STAGE, MM) do { LOAD; STAGE; WAIT_V(10); BAR; SCHED; MM; BAR; SCHED; } while (0)
#define RTC(t) asm volatile("s_memrealtime %0\n\ts_waitcnt lgkmcnt(0)" : "=s"(t) :: "memory")
    const unsigned CHK = COMM_ON ? ((1u << 2) | (1u << 9) | (1u << 16) | (1u << 23)) : 0u;
    const unsigned mP1 = __builtin_amdgcn_readfirstlane(pusher ? 0xFFFu : 0u);
    const unsigned mP2 = __builtin_amdgcn_readfirstlane(wid == 0 ? (CHK << 1) : 0u);
    const unsigned mP5 = __builtin_amdgcn_readfirstlane(pusher ? 0x1111u : (wid == 0 ? (CHK | (INV_MODE == 4 ? 2u : 0u)) : 0u));
    const unsigned mP7 = __builtin_amdgcn_readfirstlane((pusher ? 0xFFFu : 0u) | 0x80000000u | ((THR_N && wid == 0) ? (1u << 29) : 0u));
    unsigned vAlo, vAhi, vSoB;
    {
        const int v = lane - 4;
        const int g = (v * 37) >> 8;
        const bool own = lane < 4;
        const int ch = own ? rank : ((rank + 1 + (v - g * 7)) & 7);
        const int sub = own ? lane : (g & 3);
        const unsigned long long ab = (unsigned long long)(own ? xown : stage + (size_t)ch * CHUNK_BYTES) + (unsigned)sub * 256u;
        vAlo = (unsigned)ab; vAhi = (unsigned)(ab >> 32) & 0xffffu;
        vSoB = (unsigned)(ch * KC + sub * 128) * WROW;
    }
    auto mkA = [&](int pr) { const unsigned lo = __builtin_amdgcn_readlane(vAlo, pr), hi = __builtin_amdgcn_readlane(vAhi, pr);
        return __builtin_amdgcn_make_buffer_rsrc((void*)(((unsigned long long)hi << 32) | lo), 0, pr < 32 ? CHUNK_BYTES : 0u, 0x00020000); };
    auto rA2 = mkA(1);
    unsigned szW2 = W_BYTES, soB2 = __builtin_amdgcn_readlane(vSoB, 1);
    for (int u = 0; u < 32; ++u) {
        STAMP(3 + u);
        if (u == 4) STAMPC(44);
        const auto rW2 = __builtin_amdgcn_make_buffer_rsrc((void*)p.w, 0, szW2, 0x00020000);
        const unsigned bitu = 1u << u;
#define POFF ((unsigned)(u & 3) * 4u * AROW + (unsigned)(1 + (u >> 2)) * 256u)
#define GCHK ((u * 37) >> 8)
#define HK_P1 do { if (mP1 & bitu) asm volatile("global_load_dwordx4 v[232:235], %0, %1" :: "v"(voffP), "s"(psrc + POFF) : "memory", "v232", "v233", "v234", "v235"); } while (0)
#define HK_P7 do { if (mP7 & bitu) { \
            if (u == 31) { asm volatile("global_load_dword v240, %0, %1 sc1" :: "v"(vzero), "s"(myws + WS_TMO) : "memory", "v240"); \
                if (THR_N) asm volatile("global_load_dword v241, %0, %1 sc1" :: "v"(vzero), "s"(myws + WS_FIN) : "memory", "v241"); } \
            else if (THR_N && u == 29) { if (lane == 0) asm volatile("global_atomic_add %0, %1, %2 sc1" :: "v"(vzero), "v"(1u), "s"(myws + WS_FIN) : "memory"); } \
            else PSTORE("v[232:235]", pdst + POFF); } } while (0)
#define FLAG_LOAD "global_load_dwordx4 v[236:239], %0, %1 sc0 sc1"
#define HK_P5 do { if (mP5 & bitu) { if (wid != 0) { PFLAG(u >> 2); } else if (u == 1) asm volatile("buffer_inv sc0 sc1" ::: "memory"); else \
            asm volatile(FLAG_LOAD :: "v"(FLAG_VOFF), "s"(myws + WS_FLAGS + GCHK * 1024) : "memory", "v236", "v237", "v238", "v239"); } } while (0)
#define FLAG_MIN(m) asm volatile("v_min3_u32 %0, v236, v237, v238\n\tv_min_u32 %0, %0, v239" : "=&v"(m) :: "memory")
#define HK_P2 do { if (mP2 & bitu) { \
            unsigned fm; FLAG_MIN(fm); \
            if (!__all(lane >= 56 || fm >= call)) { \
                STAMPX_BEGIN; \
                unsigned long long t0, t1; RTC(t0); \
                for (;;) { \
                    unsigned tm; \
                    asm volatile(FLAG_LOAD "\n\tglobal_load_dword v240, %2, %3 sc1\n\ts_waitcnt vmcnt(0)" \
                                 :: "v"(FLAG_VOFF), "s"(myws + WS_FLAGS + GCHK * 1024), "v"(vzero), "s"(myws + WS_TMO) : "memory", "v236", "v237", "v238", "v239", "v240"); \
                    FLAG_MIN(fm); \
                    if (__all(lane >= 56 || fm >= call)) break; \
                    asm volatile("v_mov_b32 %0, v240" : "=v"(tm) :: "memory"); \
                    RTC(t1); \
                    if ((unsigned)__builtin_amdgcn_readfirstlane(tm) == call || t1 - t0 > DEADLINE_TICKS) { \
                        if (lane == 0) asm volatile("global_store_dword %0, %1, %2 sc1\n\ts_waitcnt vmcnt(0)" :: "v"(vzero), "v"(vcall), "s"(myws + WS_TMO) : "memory"); \
                        break; } \
                    __builtin_amdgcn_s_sleep(2); \
                } \
                STAMPX_END(GCHK); \
            } \
            if (INV_MODE == 2) asm volatile("buffer_inv sc0 sc1\n\ts_waitcnt vmcnt(0)" ::: "memory"); } } while (0)
#define HK_NEXT do { rA = rA2; rA2 = mkA(u + 2); szW2n = (u + 2 < 32) ? W_BYTES : 0u; soB2n = __builtin_amdgcn_readlane(vSoB, u + 2); } while (0)
#define MMAN(ai, bj, At, Bt) do { __builtin_amdgcn_s_setprio(1); HK_NEXT; MMAK(ai, bj, At, Bt, 0); MMAK(ai, bj, At, Bt, 1); \
            _Pragma("unroll") for (int i_ = 0; i_ < 16; ++i_) { __builtin_amdgcn_sched_group_barrier(0x8, 1, 0); __builtin_amdgcn_sched_group_barrier(0x6, 2, 0); } \
            __builtin_amdgcn_s_setprio(0); } while (0)
        unsigned szW2n, soB2n;
        PH(LDA(At, 0, 0), STAGE_A(1, 1, rA, 128u), MMAH(0, 0, At, B0, HK_P1));
        PH(LDB(B1, 0, 1), STAGE_B(0, 0, rW2, soB2), MMAH(0, 1, At, B1, HK_P2));
        PH(LDA(At, 0, 1), STAGE_A(0, 0, rA2, 0u), MMA(1, 0, At, B0));
        PH4(LDB(B0, 1, 0), STAGE_B(0, 1, rW2, soB2), MMA(1, 1, At, B1));
        PH(LDA(At, 1, 0), STAGE_A(0, 1, rA2, 0u), MMAH(0, 0, At, B0, HK_P5));
        PH(LDB(B1, 1, 1), STAGE_B(1, 0, rW2, soB2 + 64u * WROW), MMA(0, 1, At, B1));
        PH(LDA(At, 1, 1), STAGE_A(1, 0, rA2, 128u), MMAH(1, 0, At, B0, HK_P7));
        PH4(LDB(B0, 0, 0), STAGE_B(1, 1, rW2, soB2 + 64u * WROW), MMAN(1, 1, At, B1));
        szW2 = szW2n; soB2 = soB2n;
    }
    STAMP(35);
    STAMPC(45);
    WAIT_V(0); WAIT_L(0);
    if (wr == 0) BAR;

    {
        unsigned tmo_val, fin_val = 0;
        asm volatile("v_mov_b32 %0, v240" : "=v"(tmo_val) :: "memory");
        if (THR_N) asm volatile("v_mov_b32 %0, v241" : "=v"(fin_val) :: "memory");
        int tz = threadIdx.x; asm volatile("" : "+v"(tz));
        const int l2 = tz & 63, fr2 = l2 & 15, fq2 = l2 >> 4;
        const bool bad = (tmo_val == call);
        const unsigned fin = (unsigned)__builtin_amdgcn_readfirstlane(fin_val) & 255u;
        const bool thr = THR_N && fin != 0 && fin < THR_N;
        char* obase = p.out + (size_t)(pm * 256 + wr * 64 + fr2) * WROW + (size_t)pn * 512 + wc * 64 + ((fq2 & 1) * 32 + (fq2 >> 1) * 16);
#pragma unroll
        for (int ai = 0; ai < 2; ++ai)
#pragma unroll
            for (int m = 0; m < 4; ++m) {
                if (THR_N && thr) __builtin_amdgcn_s_sleep(THR_SLEEP);
                char* rowp = obase + (size_t)(ai * 128 + m * 16) * WROW;
#pragma unroll
                for (int bj = 0; bj < 2; ++bj) {
                    const f32x4 v0 = acc[ai][bj][m][0], v1 = acc[ai][bj][m][1];
                    unsigned a0 = relu_pk(cvt_pk_bf16(v0[0], v0[1])), a1 = relu_pk(cvt_pk_bf16(v0[2], v0[3]));
                    unsigned b0 = relu_pk(cvt_pk_bf16(v1[0], v1[1])), b1 = relu_pk(cvt_pk_bf16(v1[2], v1[3]));
                    auto r0 = __builtin_amdgcn_permlane16_swap(a0, b0, false, false);
                    auto r1 = __builtin_amdgcn_permlane16_swap(a1, b1, false, false);
                    u32x4 o; o.x = r0[0]; o.y = r1[0]; o.z = r0[1]; o.w = r1[1];
                    if (bad) o = (u32x4){0x7fc07fc0u, 0x7fc07fc0u, 0x7fc07fc0u, 0x7fc07fc0u};
                    asm volatile("global_store_dwordx4 %0, %1, off " OUT_ST "\n\ts_nop 1" :: "v"(rowp + bj * 256), "v"(o) : "memory");
                }
            }
    }
    if (tid == 0) __hip_atomic_fetch_add((GAS unsigned*)(myws + WS_CNT), 1u, __ATOMIC_RELAXED, __HIP_MEMORY_SCOPE_AGENT);
}

extern "C" void kernel_launch(int rank, int nranks, ncclComm_t comm,
                              hipStream_t stream,
                              void* const* d_in, const int* in_sizes, int n_in,
                              void* d_out, int out_size,
                              const int* in_dtypes, int out_dtype,
                              void* const* d_out_all,
                              void* const* d_ws_all, size_t ws_bytes) {
    (void)comm; (void)in_dtypes; (void)out_dtype; (void)d_out_all;
    if (nranks != NR || n_in < 2 || rank < 0 || rank >= NR || ws_bytes < WS_NEED ||
        in_sizes[0] != KDIM * KC || in_sizes[1] != KDIM * NDIM || out_size != MLOC * NDIM) {
        fprintf(stderr, "kernel_launch: unexpected problem shape\n");
        return;
    }
    static bool first = true;
    if (first) {
        HIP_CHECK(hipFuncSetAttribute((const void*)a2a_gemm_kernel, hipFuncAttributeMaxDynamicSharedMemorySize, LDS_BYTES));
        first = false;
    }
    Params p;
    p.x = (const char*)d_in[0];
    p.w = (const char*)d_in[1];
    p.out = (char*)d_out;
    for (int i = 0; i < NR; ++i) p.ws[i] = (char*)d_ws_all[i];
    p.rank = rank;
    a2a_gemm_kernel<<<GRID, 512, LDS_BYTES, stream>>>(p);
    HIP_CHECK(hipGetLastError());
}
```

```cpp
#include <hip/hip_runtime.h>
#include <rccl/rccl.h>
#include <cstdio>
#include <cstdint>

#define HIP_CHECK(expr) do { hipError_t _e = (expr); if (_e != hipSuccess) { \
    fprintf(stderr, "HIP error %s at %s:%d\n", hipGetErrorString(_e), __FILE__, __LINE__); \
    return; } } while (0)

#define UNLIKELY(x) __builtin_expect(!!(x), 0)
#define LAS __attribute__((address_space(3)))
#define GAS __attribute__((address_space(1)))
typedef short s16x8 __attribute__((ext_vector_type(8)));
typedef short s16x4 __attribute__((ext_vector_type(4)));
typedef short s16x2 __attribute__((ext_vector_type(2)));
typedef __bf16 bf16x8 __attribute__((ext_vector_type(8)));
typedef float f32x4 __attribute__((ext_vector_type(4)));
typedef unsigned u32x4 __attribute__((ext_vector_type(4)));

constexpr int NR = 8;
constexpr int MLOC = 512;
constexpr int KDIM = 4096;
constexpr int NDIM = 32768;
constexpr int KC = KDIM / NR;
constexpr unsigned CHUNK_BYTES = MLOC * KC * 2;
constexpr int HTB = 16384;
constexpr int LDS_BYTES = 8 * HTB;
constexpr unsigned W_BYTES = (unsigned)KDIM * NDIM * 2u;
constexpr unsigned WROW = NDIM * 2;
constexpr unsigned AROW = KC * 2;
constexpr int GRID = 256;
constexpr size_t WS_CNT = 0;
constexpr size_t WS_TMO = 128;
constexpr size_t WS_FIN = 256;
constexpr size_t WS_FLAGS = 4096;
constexpr size_t WS_STAGE = 1u << 20;
constexpr size_t SLOT_BYTES = (size_t)NR * CHUNK_BYTES;
constexpr size_t WS_NEED = WS_STAGE + 2 * SLOT_BYTES;
constexpr unsigned long long DEADLINE_TICKS = 40000000ull;

#define STAMP(i) do { } while (0)
#define STAMPC(i) do { } while (0)
#define STAMPX_BEGIN do { } while (0)
#define STAMPX_END(g) do { } while (0)
#define W_AUX 0
#define OUT_ST "sc1 nt"
#define COMM_ON true
#define THR_N 0
#define THR_SLEEP 8
#define INV_MODE 4
struct Params {
    const char* x;
    const char* w;
    char* out;
    char* ws[NR];
    int rank;
};

__device__ __forceinline__ unsigned cvt_pk_bf16(float lo, float hi) {
    unsigned r; asm("v_cvt_pk_bf16_f32 %0, %1, %2" : "=v"(r) : "v"(lo), "v"(hi)); return r;
}
__device__ __forceinline__ unsigned relu_pk(unsigned v) {
    s16x2 a = __builtin_bit_cast(s16x2, v);
    s16x2 z = {0, 0};
    return __builtin_bit_cast(unsigned, __builtin_elementwise_max(a, z));
}

template <class T> __device__ __forceinline__ T* uni(T* ptr) {
    const unsigned long long v = (unsigned long long)ptr;
    const unsigned lo = __builtin_amdgcn_readfirstlane((unsigned)v), hi = __builtin_amdgcn_readfirstlane((unsigned)(v >> 32));
    return (T*)(((unsigned long long)hi << 32) | lo);
}

__global__ __launch_bounds__(512, 2) void a2a_gemm_kernel(Params p) {
    extern __shared__ __attribute__((aligned(16))) char smem_[];
    LAS char* lds = (LAS char*)smem_;
    const int tid = threadIdx.x;
    const int wid = __builtin_amdgcn_readfirstlane(tid >> 6);
    const int lane = tid & 63, wr = wid >> 2, wc = wid & 3, fr = lane & 15, fq = lane >> 4;
    const int bid = blockIdx.x;
    const int xcd = bid & 7, jj = bid >> 3;
    const int pm = jj & 1, pn = xcd * 16 + (jj >> 1);
    const int rank = p.rank;
    char* const myws = uni(p.ws[rank]);
    STAMP(0);

    const unsigned vzero = 0;
    asm volatile("global_load_dword v239, %0, %1 sc1" :: "v"(vzero), "s"(myws + WS_CNT) : "memory", "v239");

    const bool pusher = COMM_ON && (wid == 1) && (bid < 224);
    const unsigned dest = (unsigned)(rank + 1 + (bid >> 5)) & 7u;
    const unsigned pj = (unsigned)bid & 31u;
    const unsigned voffP = (unsigned)(lane >> 4) * AROW + (unsigned)(lane & 15) * 16u;
    const char* const psrc = uni(p.x + (size_t)(dest * MLOC + pj * 16u) * AROW);
    char* const dws = uni(p.ws[dest]);
    if (pusher) {
        asm volatile("global_load_dwordx4 v[240:243], %0, %1" :: "v"(voffP), "s"(psrc) : "memory", "v240", "v241", "v242", "v243");
        asm volatile("global_load_dwordx4 v[244:247], %0, %1" :: "v"(voffP), "s"(psrc + 4 * AROW) : "memory", "v244", "v245", "v246", "v247");
        asm volatile("global_load_dwordx4 v[248:251], %0, %1" :: "v"(voffP), "s"(psrc + 8 * AROW) : "memory", "v248", "v249", "v250", "v251");
        asm volatile("global_load_dwordx4 v[252:255], %0, %1" :: "v"(voffP), "s"(psrc + 12 * AROW) : "memory", "v252", "v253", "v254", "v255");
    }

    const unsigned voffA = (unsigned)(lane >> 3) * AROW + (unsigned)(((lane & 7) ^ (lane >> 3)) * 16);
    const unsigned voffB = (unsigned)(((lane >> 3) & 1) * 8 + (lane >> 4)) * WROW + (unsigned)(((lane & 7) ^ ((lane >> 4) << 1)) * 16);
    const unsigned sA_w = (unsigned)(pm * 256 + wid * 8) * AROW;
    const unsigned sB_w = (unsigned)(((wid >> 2) & 1) * 16 + ((wid >> 1) & 1) * 4) * WROW + (unsigned)(wid & 1) * 128u + (unsigned)pn * 512u;
    const unsigned ldsw = (unsigned)wid * 1024u;
    const int arow = wr * 64 + fr;
    const unsigned aoff0 = (unsigned)arow * 128u + (unsigned)((fq ^ (fr & 7)) * 16);
    const unsigned aoff1 = aoff0 ^ 64u;
    const int q = (lane & 15) >> 2, pp = lane & 3;
    const unsigned bcom = (unsigned)(fq >> 1) * 4096u + (unsigned)(wc >> 1) * 1024u + (unsigned)q * 256u + (unsigned)(fq & 1) * 128u + (unsigned)pp * 8u;
    const unsigned ldsbase = (unsigned)(size_t)lds;
    const unsigned boff0 = ldsbase + 4 * HTB + bcom + (unsigned)(((2 * (wc & 1) + 0) ^ q) * 32);
    const unsigned boff1 = ldsbase + 4 * HTB + bcom + (unsigned)(((2 * (wc & 1) + 1) ^ q) * 32);

#define SA(b, h) (((b) * 2 + (h)) * HTB)
#define SB(b, h) (((b) * 2 + (h)) * HTB)
#define STAGE_A(b, h, rs, so) do { \
        __builtin_amdgcn_raw_ptr_buffer_load_lds(rs, (LAS void*)(lds + SA(b, h) + ldsw), 16, voffA, (so) + sA_w + (h) * 128u * AROW, 0, 0); \
        __builtin_amdgcn_raw_ptr_buffer_load_lds(rs, (LAS void*)(lds + SA(b, h) + ldsw + 8192), 16, voffA, (so) + sA_w + (h) * 128u * AROW + 64u * AROW, 0, 0); } while (0)
#define STAGE_B(b, h, rs, so) do { \
        __builtin_amdgcn_raw_ptr_buffer_load_lds(rs, (LAS void*)(lds + 4 * HTB + SB(b, h) + ldsw), 16, voffB, (so) + sB_w + (h) * 256u, 0, W_AUX); \
        __builtin_amdgcn_raw_ptr_buffer_load_lds(rs, (LAS void*)(lds + 4 * HTB + SB(b, h) + ldsw + 8192), 16, voffB, (so) + sB_w + (h) * 256u + 32u * WROW, 0, W_AUX); } while (0)
#define LDA(dst, b, h) do { _Pragma("unroll") for (int m = 0; m < 4; ++m) { \
        dst[m][0] = *(const LAS s16x8*)(lds + SA(b, h) + aoff0 + m * 2048); \
        dst[m][1] = *(const LAS s16x8*)(lds + SA(b, h) + aoff1 + m * 2048); } } while (0)
#define TRR(dst, base, off) asm volatile("ds_read_b64_tr_b16 %0, %1 offset:%2" : "=v"(dst) : "v"(base), "i"(off) : "memory")
#define LDB(dst, b, h) do { _Pragma("unroll") for (int k = 0; k < 2; ++k) { \
        { s16x4 lo, hi; TRR(lo, boff0, SB(b, h) + k * 8192); TRR(hi, boff0, SB(b, h) + k * 8192 + 2048); dst[0][k] = __builtin_shufflevector(lo, hi, 0, 1, 2, 3, 4, 5, 6, 7); } \
        { s16x4 lo, hi; TRR(lo, boff1, SB(b, h) + k * 8192); TRR(hi, boff1, SB(b, h) + k * 8192 + 2048); dst[1][k] = __builtin_shufflevector(lo, hi, 0, 1, 2, 3, 4, 5, 6, 7); } } } while (0)
#define MF1(ai, bj, At, Bt, m, n, k) acc[ai][bj][m][n] = __builtin_amdgcn_mfma_f32_16x16x32_bf16(__builtin_bit_cast(bf16x8, Bt[n][k]), __builtin_bit_cast(bf16x8, At[m][k]), acc[ai][bj][m][n], 0, 0, 0)
#define MMA2(ai, At, Ba, Bb) do { __builtin_amdgcn_s_setprio(1); \
        _Pragma("unroll") for (int k = 0; k < 2; ++k) { \
            _Pragma("unroll") for (int m = 0; m < 4; ++m) _Pragma("unroll") for (int n = 0; n < 2; ++n) MF1(ai, 0, At, Ba, m, n, k); \
            _Pragma("unroll") for (int m = 0; m < 4; ++m) _Pragma("unroll") for (int n = 0; n < 2; ++n) MF1(ai, 1, At, Bb, m, n, k); } \
        __builtin_amdgcn_s_setprio(0); } while (0)
#define MMAK2(ai, At, Ba, Bb, k) do { \
            _Pragma("unroll") for (int m = 0; m < 4; ++m) _Pragma("unroll") for (int n = 0; n < 2; ++n) MF1(ai, 0, At, Ba, m, n, k); \
            _Pragma("unroll") for (int m = 0; m < 4; ++m) _Pragma("unroll") for (int n = 0; n < 2; ++n) MF1(ai, 1, At, Bb, m, n, k); } while (0)
#define MMA2H(ai, At, Ba, Bb, HOOK) do { __builtin_amdgcn_s_setprio(1); MMAK2(ai, At, Ba, Bb, 0); SCHED; HOOK; SCHED; MMAK2(ai, At, Ba, Bb, 1); __builtin_amdgcn_s_setprio(0); } while (0)
#define MMA2N(ai, At, Ba, Bb, SCALARS) do { __builtin_amdgcn_s_setprio(1); SCALARS; MMAK2(ai, At, Ba, Bb, 0); MMAK2(ai, At, Ba, Bb, 1); \
            _Pragma("unroll") for (int i_ = 0; i_ < 32; ++i_) { __builtin_amdgcn_sched_group_barrier(0x8, 1, 0); __builtin_amdgcn_sched_group_barrier(0x6, 1, 0); } \
            __builtin_amdgcn_s_setprio(0); } while (0)
#define WAIT_V(n) asm volatile("s_waitcnt vmcnt(" #n ")" ::: "memory")
#define WAIT_L(n) asm volatile("s_waitcnt lgkmcnt(" #n ")" ::: "memory")
#define BAR __builtin_amdgcn_s_barrier()
#define SCHED __builtin_amdgcn_sched_barrier(0)
#define PSTORE(reg, base) asm volatile("global_store_dwordx4 %0, " reg ", %1 sc0 sc1\n\ts_nop 1" :: "v"(voffP), "s"(base) : "memory")
#define RTC(t) asm volatile("s_memrealtime %0\n\ts_waitcnt lgkmcnt(0)" : "=s"(t) :: "memory")

    f32x4 acc[2][2][4][2];
#pragma unroll
    for (int a = 0; a < 2; ++a)
#pragma unroll
        for (int b = 0; b < 2; ++b)
#pragma unroll
            for (int m = 0; m < 4; ++m)
#pragma unroll
                for (int n = 0; n < 2; ++n) acc[a][b][m][n] = (f32x4){0.f, 0.f, 0.f, 0.f};
    s16x8 At[4][2], B0a[2][2], B0b[2][2], B1[2][2];

    const auto rW = __builtin_amdgcn_make_buffer_rsrc((void*)p.w, 0, W_BYTES, 0x00020000);
    const char* const xown = uni(p.x + (size_t)rank * CHUNK_BYTES);
    auto rA = __builtin_amdgcn_make_buffer_rsrc((void*)xown, 0, CHUNK_BYTES, 0x00020000);
    const unsigned soB0 = (unsigned)(rank * KC) * WROW;

    STAGE_B(0, 0, rW, soB0); STAGE_A(0, 0, rA, 0u); STAGE_B(0, 1, rW, soB0);
    STAGE_B(1, 0, rW, soB0 + 64u * WROW); STAGE_A(0, 1, rA, 0u);
    WAIT_V(10);
    unsigned cntv;
    asm volatile("v_mov_b32 %0, v239" : "=v"(cntv) :: "memory");
    const unsigned call = (unsigned)__builtin_amdgcn_readfirstlane(cntv) / GRID + 1u;
    const size_t slot_off = WS_STAGE + (size_t)(call & 1u) * SLOT_BYTES;
    const char* const stage = uni(myws + slot_off);
    char* const pdst = uni(dws + slot_off + (size_t)rank * CHUNK_BYTES + (size_t)(pj * 16u) * AROW);
    char* const pflag = uni(dws + WS_FLAGS + (size_t)rank * 128 + pj * 4u);
    STAMP(1);
    if (pusher) PSTORE("v[240:243]", pdst);
    STAMP(38);
    STAGE_A(1, 0, rA, 128u); STAGE_B(1, 1, rW, soB0 + 64u * WROW);
    STAMP(39);
    if (pusher) WAIT_V(9); else WAIT_V(8);
    STAMP(47);
    BAR;
    LDB(B0a, 0, 0);
    if (wr == 1) BAR;
    STAMP(2);

    unsigned vAlo, vAhi, vSoB;
    {
        const int v = lane - 4;
        const int g = (v * 37) >> 8;
        const bool own = lane < 4;
        const int ch = own ? rank : ((rank + 1 + (v - g * 7)) & 7);
        const int sub = own ? lane : (g & 3);
        const unsigned long long ab = (unsigned long long)(own ? xown : stage + (size_t)ch * CHUNK_BYTES) + (unsigned)sub * 256u;
        vAlo = (unsigned)ab; vAhi = (unsigned)(ab >> 32) & 0xffffu;
        vSoB = (unsigned)(ch * KC + sub * 128) * WROW;
    }
    auto mkA = [&](int pr) { const unsigned lo = __builtin_amdgcn_readlane(vAlo, pr), hi = __builtin_amdgcn_readlane(vAhi, pr);
        return __builtin_amdgcn_make_buffer_rsrc((void*)(((unsigned long long)hi << 32) | lo), 0, CHUNK_BYTES, 0x00020000); };
    auto rA2 = mkA(1);
    unsigned soB2 = __builtin_amdgcn_readlane(vSoB, 1);

    const unsigned CHK = COMM_ON ? ((1u << 2) | (1u << 9) | (1u << 16) | (1u << 23)) : 0u;
    const unsigned PUSH = (3u << 1) | (3u << 9) | (3u << 16);
    const unsigned FLG = (1u << 1) | (1u << 3) | (1u << 11) | (1u << 18);
    const unsigned mA = __builtin_amdgcn_readfirstlane(pusher ? (PUSH | 1u) : (wid == 0 ? (CHK << 1) : 0u));
    const unsigned mA10 = __builtin_amdgcn_readfirstlane(pusher ? (PUSH << 1) : 0u);
    const unsigned mB = __builtin_amdgcn_readfirstlane(pusher ? (PUSH | FLG | 1u) : 0u);
    const unsigned mC = __builtin_amdgcn_readfirstlane((pusher ? (PUSH | 1u) : (wid == 0 ? (CHK | ((COMM_ON && INV_MODE == 4) ? 2u : 0u)) : 0u)) | (1u << 30));
    const unsigned mD = __builtin_amdgcn_readfirstlane(pusher ? PUSH : 0u);
    const unsigned vcall = call;
#define PFLAG(g) do { if (lane == 0) asm volatile("global_store_dword %0, %1, %2 sc0 sc1" :: "v"(vzero), "v"(vcall), "s"(pflag + (g) * 1024) : "memory"); } while (0)
#define FLAG_VOFF ((unsigned)(((rank + 1 + (lane >> 3)) & 7) * 128 + (lane & 7) * 16))
#define FLAG_LOAD "global_load_dwordx4 v[244:247], %0, %1 sc0 sc1"
#define FLAG_MIN(m) asm volatile("v_min3_u32 %0, v244, v245, v246\n\tv_min_u32 %0, %0, v247" : "=&v"(m) :: "memory")
#define PGRP (u < 9 ? 1 : (u < 16 ? 2 : 3))
#define POFF(j) ((unsigned)(2 * (u - (u < 9 ? 1 : (u < 16 ? 9 : 16))) + (j)) * 4u * AROW + (unsigned)PGRP * 256u)
#define GCHK ((u * 37) >> 8)
#define HK_A do { if (UNLIKELY(mA & bitu)) { if (wid != 0) { if (u == 0) PSTORE("v[244:247]", pdst + 4 * AROW); else \
            asm volatile("global_load_dwordx4 v[240:243], %0, %1" :: "v"(voffP), "s"(psrc + POFF(0)) : "memory", "v240", "v241", "v242", "v243"); } \
        else { \
            unsigned fm; FLAG_MIN(fm); \
            if (!__all(lane >= 56 || fm >= call)) { \
                STAMPX_BEGIN; \
                unsigned long long t0, t1; RTC(t0); \
                for (;;) { \
                    unsigned tm; \
                    asm volatile(FLAG_LOAD "\n\tglobal_load_dword v248, %2, %3 sc1\n\ts_waitcnt vmcnt(0)" \
                                 :: "v"(FLAG_VOFF), "s"(myws + WS_FLAGS + GCHK * 1024), "v"(vzero), "s"(myws + WS_TMO) : "memory", "v244", "v245", "v246", "v247", "v248"); \
                    FLAG_MIN(fm); \
                    if (__all(lane >= 56 || fm >= call)) break; \
                    asm volatile("v_mov_b32 %0, v248" : "=v"(tm) :: "memory"); \
                    RTC(t1); \
                    if ((unsigned)__builtin_amdgcn_readfirstlane(tm) == call || t1 - t0 > DEADLINE_TICKS) { \
                        if (lane == 0) asm volatile("global_store_dword %0, %1, %2 sc1\n\ts_waitcnt vmcnt(0)" :: "v"(vzero), "v"(vcall), "s"(myws + WS_TMO) : "memory"); \
                        break; } \
                    __builtin_amdgcn_s_sleep(2); \
                } \
                STAMPX_END(GCHK); \
            } } } } while (0)
#define HK_B do { if (UNLIKELY(mB & bitu)) { if (u == 0) PSTORE("v[248:251]", pdst + 8 * AROW); else { \
            if (FLG & bitu) PFLAG(u == 1 ? 0 : PGRP); \
            if (PUSH & bitu) asm volatile("global_load_dwordx4 v[252:255], %0, %1" :: "v"(voffP), "s"(psrc + POFF(1)) : "memory", "v252", "v253", "v254", "v255"); } } } while (0)
#define HK_C do { if (UNLIKELY(mC & bitu)) { \
            if (u == 30) asm volatile("global_load_dword v248, %0, %1 sc1" :: "v"(vzero), "s"(myws + WS_TMO) : "memory", "v248"); \
            else if (wid != 0) { if (u == 0) PSTORE("v[252:255]", pdst + 12 * AROW); else PSTORE("v[240:243]", pdst + POFF(0)); } \
            else if (u == 1) asm volatile("buffer_inv sc0 sc1" ::: "memory"); \
            else asm volatile(FLAG_LOAD :: "v"(FLAG_VOFF), "s"(myws + WS_FLAGS + GCHK * 1024) : "memory", "v244", "v245", "v246", "v247"); } } while (0)
#define HK_D do { if (UNLIKELY(mD & bitu)) PSTORE("v[252:255]", pdst + POFF(1)); } while (0)
#define PHASE(L1, L2, S1, S2, WAITV, MM) do { L1; L2; SCHED; S1; S2; WAITV; WAIT_L(0); BAR; SCHED; MM; BAR; SCHED; } while (0)
#define NEXT_PAIR do { rA = rA2; rA2 = mkA((u + 2) & 31); soB3 = __builtin_amdgcn_readlane(vSoB, (u + 2) & 31); } while (0)
    int u = 0;
    for (int seg = 0; seg < 5; ++seg) {
        const int nh = (0x12334 >> (4 * seg)) & 15, np = (0x05445 >> (4 * seg)) & 15;
        for (int i = 0; i < nh; ++i, ++u) {
            if (u == 4) { STAMPC(44); STAMP(7); }
            const unsigned bitu = 1u << u;
            unsigned soB3;
            PHASE(LDA(At, 0, 0), LDB(B1, 0, 1), STAGE_B(0, 0, rW, soB2), STAGE_A(1, 1, rA, 128u),
                  do { WAIT_V(10); if (!UNLIKELY(mA10 & bitu)) WAIT_V(8); } while (0), MMA2H(0, At, B0a, B1, HK_A));
            PHASE(LDA(At, 0, 1), LDB(B0b, 1, 0), STAGE_A(0, 0, rA2, 0u), STAGE_B(0, 1, rW, soB2), WAIT_V(8), MMA2H(1, At, B0a, B1, HK_B));
            PHASE(LDA(At, 1, 0), LDB(B1, 1, 1), STAGE_B(1, 0, rW, soB2 + 64u * WROW), STAGE_A(0, 1, rA2, 0u), WAIT_V(8), MMA2H(0, At, B0b, B1, HK_C));
            PHASE(LDA(At, 1, 1), LDB(B0a, 0, 0), STAGE_A(1, 0, rA2, 128u), STAGE_B(1, 1, rW, soB2 + 64u * WROW), do { WAIT_V(8); NEXT_PAIR; } while (0),
                  MMA2H(1, At, B0b, B1, HK_D));
            soB2 = soB3;
        }
        for (int i = 0; i < np; ++i, ++u) {
            if (u == 4) { STAMPC(44); STAMP(7); }
            unsigned soB3;
            PHASE(LDA(At, 0, 0), LDB(B1, 0, 1), STAGE_B(0, 0, rW, soB2), STAGE_A(1, 1, rA, 128u), WAIT_V(8), MMA2(0, At, B0a, B1));
            PHASE(LDA(At, 0, 1), LDB(B0b, 1, 0), STAGE_A(0, 0, rA2, 0u), STAGE_B(0, 1, rW, soB2), WAIT_V(8), MMA2(1, At, B0a, B1));
            PHASE(LDA(At, 1, 0), LDB(B1, 1, 1), STAGE_B(1, 0, rW, soB2 + 64u * WROW), STAGE_A(0, 1, rA2, 0u), WAIT_V(8), MMA2(0, At, B0b, B1));
            PHASE(LDA(At, 1, 1), LDB(B0a, 0, 0), STAGE_A(1, 0, rA2, 128u), STAGE_B(1, 1, rW, soB2 + 64u * WROW), WAIT_V(8),
                  MMA2N(1, At, B0b, B1, NEXT_PAIR));
            soB2 = soB3;
        }
    }
    STAMP(34); STAMPC(46);

    int tz = threadIdx.x; asm volatile("" : "+v"(tz));
    const int l2 = tz & 63, fr2 = l2 & 15, fq2 = l2 >> 4;
    char* const obase = p.out + (size_t)(pm * 256 + wr * 64 + fr2) * WROW + (size_t)pn * 512 + wc * 64 + ((fq2 & 1) * 32 + (fq2 >> 1) * 16);
    bool bad;
#define EPI(ai) do { _Pragma("unroll") for (int m = 0; m < 4; ++m) { \
            char* rowp = obase + (size_t)((ai) * 128 + m * 16) * WROW; \
            _Pragma("unroll") for (int bj = 0; bj < 2; ++bj) { \
                const f32x4 v0 = acc[ai][bj][m][0], v1 = acc[ai][bj][m][1]; \
                unsigned a0 = relu_pk(cvt_pk_bf16(v0[0], v0[1])), a1 = relu_pk(cvt_pk_bf16(v0[2], v0[3])); \
                unsigned b0 = relu_pk(cvt_pk_bf16(v1[0], v1[1])), b1 = relu_pk(cvt_pk_bf16(v1[2], v1[3])); \
                auto r0 = __builtin_amdgcn_permlane16_swap(a0, b0, false, false); \
                auto r1 = __builtin_amdgcn_permlane16_swap(a1, b1, false, false); \
                u32x4 o; o.x = r0[0]; o.y = r1[0]; o.z = r0[1]; o.w = r1[1]; \
                if (bad) o = (u32x4){0x7fc07fc0u, 0x7fc07fc0u, 0x7fc07fc0u, 0x7fc07fc0u}; \
                asm volatile("global_store_dwordx4 %0, %1, off " OUT_ST "\n\ts_nop 1" :: "v"(rowp + bj * 256), "v"(o) : "memory"); } } } while (0)
    PHASE(LDA(At, 0, 0), LDB(B1, 0, 1), (void)0, STAGE_A(1, 1, rA, 128u), WAIT_V(6), MMA2(0, At, B0a, B1));
    { unsigned tm; asm volatile("v_mov_b32 %0, v248" : "=v"(tm) :: "memory"); bad = ((unsigned)__builtin_amdgcn_readfirstlane(tm) == call); }
    PHASE(LDA(At, 0, 1), LDB(B0b, 1, 0), (void)0, (void)0, WAIT_V(2), MMA2(1, At, B0a, B1));
    PHASE(LDA(At, 1, 0), LDB(B1, 1, 1), (void)0, (void)0, WAIT_V(0), MMA2(0, At, B0b, B1));
    LDA(At, 1, 1); SCHED;
    EPI(0);
    WAIT_L(0); BAR; SCHED;
    MMA2(1, At, B0b, B1);
    if (wr == 0) BAR;
    SCHED;
    STAMP(35);
    STAMPC(45);
    EPI(1);
    if (tid == 0) __hip_atomic_fetch_add((GAS unsigned*)(myws + WS_CNT), 1u, __ATOMIC_RELAXED, __HIP_MEMORY_SCOPE_AGENT);
}

extern "C" void kernel_launch(int rank, int nranks, ncclComm_t comm,
                              hipStream_t stream,
                              void* const* d_in, const int* in_sizes, int n_in,
                              void* d_out, int out_size,
                              const int* in_dtypes, int out_dtype,
                              void* const* d_out_all,
                              void* const* d_ws_all, size_t ws_bytes) {
    (void)comm; (void)in_dtypes; (void)out_dtype; (void)d_out_all;
    if (nranks != NR || n_in < 2 || rank < 0 || rank >= NR || ws_bytes < WS_NEED ||
        in_sizes[0] != KDIM * KC || in_sizes[1] != KDIM * NDIM || out_size != MLOC * NDIM) {
        fprintf(stderr, "kernel_launch: unexpected problem shape\n");
        return;
    }
    static bool first = true;
    if (first) {
        HIP_CHECK(hipFuncSetAttribute((const void*)a2a_gemm_kernel, hipFuncAttributeMaxDynamicSharedMemorySize, LDS_BYTES));
        first = false;
    }
    Params p;
    p.x = (const char*)d_in[0];
    p.w = (const char*)d_in[1];
    p.out = (char*)d_out;
    for (int i = 0; i < NR; ++i) p.ws[i] = (char*)d_ws_all[i];
    p.rank = rank;
    a2a_gemm_kernel<<<GRID, 512, LDS_BYTES, stream>>>(p);
    HIP_CHECK(hipGetLastError());
}
```

```cpp
#include <hip/hip_runtime.h>
#include <rccl/rccl.h>
#include <cstdio>
#include <cstdint>
#pragma clang diagnostic ignored "-Winline-asm"

#define HIP_CHECK(expr) do { hipError_t _e = (expr); if (_e != hipSuccess) { \
    fprintf(stderr, "HIP error %s at %s:%d\n", hipGetErrorString(_e), __FILE__, __LINE__); \
    return; } } while (0)

#define UNLIKELY(x) __builtin_expect(!!(x), 0)
#define LAS __attribute__((address_space(3)))
#define GAS __attribute__((address_space(1)))
typedef short s16x8 __attribute__((ext_vector_type(8)));
typedef short s16x4 __attribute__((ext_vector_type(4)));
typedef short s16x2 __attribute__((ext_vector_type(2)));
typedef __bf16 bf16x8 __attribute__((ext_vector_type(8)));
typedef float f32x4 __attribute__((ext_vector_type(4)));
typedef unsigned u32x4 __attribute__((ext_vector_type(4)));

constexpr int NR = 8;
constexpr int MLOC = 512;
constexpr int KDIM = 4096;
constexpr int NDIM = 32768;
constexpr int KC = KDIM / NR;
constexpr unsigned CHUNK_BYTES = MLOC * KC * 2;
constexpr int HTB = 16384;
constexpr int LDS_BYTES = 8 * HTB;
constexpr unsigned W_BYTES = (unsigned)KDIM * NDIM * 2u;
constexpr unsigned WROW = NDIM * 2;
constexpr unsigned AROW = KC * 2;
constexpr int GRID = 256;
constexpr size_t WS_CNT = 0;
constexpr size_t WS_TMO = 128;
constexpr size_t WS_FIN = 256;
constexpr size_t WS_FLAGS = 4096;
constexpr size_t WS_STAGE = 1u << 20;
constexpr size_t SLOT_BYTES = (size_t)NR * CHUNK_BYTES;
constexpr size_t WS_NEED = WS_STAGE + 2 * SLOT_BYTES;
constexpr unsigned long long DEADLINE_TICKS = 40000000ull;

#define STAMP(i) do { } while (0)
#define STAMPC(i) do { } while (0)
#define STAMPX_BEGIN do { } while (0)
#define STAMPX_END(g) do { } while (0)
#define W_AUX 0
#define OUT_ST "sc1 nt"
#define COMM_ON true
#define THR_N 0
#define THR_SLEEP 8
#define PREFETCH 1
#define PF_U 27
#define PF_ROLL 0
#define INV_MODE 4
#define LANE_NOW(l) asm volatile("v_mbcnt_lo_u32_b32 %0, -1, 0\n\tv_mbcnt_hi_u32_b32 %0, -1, %0" : "=v"(l))
#define VOFFP(l) ((unsigned)((l) >> 4) * AROW + (unsigned)((l) & 15) * 16u)
struct Params {
    const char* x;
    const char* w;
    char* out;
    char* ws[NR];
    int rank;
};

__device__ __forceinline__ unsigned cvt_pk_bf16(float lo, float hi) {
    unsigned r; asm("v_cvt_pk_bf16_f32 %0, %1, %2" : "=v"(r) : "v"(lo), "v"(hi)); return r;
}
__device__ __forceinline__ unsigned relu_pk(unsigned v) {
    s16x2 a = __builtin_bit_cast(s16x2, v);
    s16x2 z = {0, 0};
    return __builtin_bit_cast(unsigned, __builtin_elementwise_max(a, z));
}

template <class T> __device__ __forceinline__ T* uni(T* ptr) {
    const unsigned long long v = (unsigned long long)ptr;
    const unsigned lo = __builtin_amdgcn_readfirstlane((unsigned)v), hi = __builtin_amdgcn_readfirstlane((unsigned)(v >> 32));
    return (T*)(((unsigned long long)hi << 32) | lo);
}

__global__ __launch_bounds__(512, 2) __attribute__((amdgpu_num_vgpr(120))) void a2a_gemm_kernel(Params p) {
    extern __shared__ __attribute__((aligned(16))) char smem_[];
    LAS char* lds = (LAS char*)smem_;
    const int tid = threadIdx.x;
    const int wid = __builtin_amdgcn_readfirstlane(tid >> 6);
    const int lane = tid & 63, wr = wid >> 2, wc = wid & 3, fr = lane & 15, fq = lane >> 4;
    const int bid = blockIdx.x;
    const int xcd = bid & 7, jj = bid >> 3;
    const int pm = jj & 1, pn = xcd * 16 + (jj >> 1);
    const int rank = p.rank;
    char* const myws = uni(p.ws[rank]);
    STAMP(0);

    const unsigned vzero = 0;
    asm volatile("global_load_dword v239, %0, %1 sc1" :: "v"(vzero), "s"(myws + WS_CNT) : "memory", "v239");

    const bool pusher = COMM_ON && (wid == 1) && (bid < 224);
    const unsigned dest = (unsigned)(rank + 1 + (bid >> 5)) & 7u;
    const unsigned pj = (unsigned)bid & 31u;
    const char* const psrc = uni(p.x + (size_t)(dest * MLOC + pj * 16u) * AROW);
    char* const dws = uni(p.ws[dest]);
    if (pusher) {
        const unsigned voffP = VOFFP(lane);
        asm volatile("global_load_dwordx4 v[240:243], %0, %1" :: "v"(voffP), "s"(psrc) : "memory", "v240", "v241", "v242", "v243");
        asm volatile("global_load_dwordx4 v[244:247], %0, %1" :: "v"(voffP), "s"(psrc + 4 * AROW) : "memory", "v244", "v245", "v246", "v247");
        asm volatile("global_load_dwordx4 v[248:251], %0, %1" :: "v"(voffP), "s"(psrc + 8 * AROW) : "memory", "v248", "v249", "v250", "v251");
        asm volatile("global_load_dwordx4 v[252:255], %0, %1" :: "v"(voffP), "s"(psrc + 12 * AROW) : "memory", "v252", "v253", "v254", "v255");
    }

    const unsigned voffA = (unsigned)(lane >> 3) * AROW + (unsigned)(((lane & 7) ^ (lane >> 3)) * 16);
    const unsigned voffB = (unsigned)(((lane >> 3) & 1) * 8 + (lane >> 4)) * WROW + (unsigned)(((lane & 7) ^ ((lane >> 4) << 1)) * 16);
    const unsigned sA_w = (unsigned)(pm * 256 + wid * 8) * AROW;
    const unsigned sB_w = (unsigned)(((wid >> 2) & 1) * 16 + ((wid >> 1) & 1) * 4) * WROW + (unsigned)(wid & 1) * 128u + (unsigned)pn * 512u;
    const unsigned ldsw = (unsigned)wid * 1024u;
    const int arow = wr * 64 + fr;
    const unsigned aoff0 = (unsigned)arow * 128u + (unsigned)((fq ^ (fr & 7)) * 16);
    const unsigned aoff1 = aoff0 ^ 64u;
    const int q = (lane & 15) >> 2, pp = lane & 3;
    const unsigned bcom = (unsigned)(fq >> 1) * 4096u + (unsigned)(wc >> 1) * 1024u + (unsigned)q * 256u + (unsigned)(fq & 1) * 128u + (unsigned)pp * 8u;
    const unsigned ldsbase = (unsigned)(size_t)lds;
    const unsigned boff0 = ldsbase + 4 * HTB + bcom + (unsigned)(((2 * (wc & 1) + 0) ^ q) * 32);
    const unsigned boff1 = ldsbase + 4 * HTB + bcom + (unsigned)(((2 * (wc & 1) + 1) ^ q) * 32);

#define SA(b, h) (((b) * 2 + (h)) * HTB)
#define SB(b, h) (((b) * 2 + (h)) * HTB)
#define STAGE_A(b, h, rs, so) do { \
        __builtin_amdgcn_raw_ptr_buffer_load_lds(rs, (LAS void*)(lds + SA(b, h) + ldsw), 16, voffA, (so) + sA_w + (h) * 128u * AROW, 0, 0); \
        __builtin_amdgcn_raw_ptr_buffer_load_lds(rs, (LAS void*)(lds + SA(b, h) + ldsw + 8192), 16, voffA, (so) + sA_w + (h) * 128u * AROW + 64u * AROW, 0, 0); } while (0)
#define STAGE_B(b, h, rs, so) do { \
        __builtin_amdgcn_raw_ptr_buffer_load_lds(rs, (LAS void*)(lds + 4 * HTB + SB(b, h) + ldsw), 16, voffB, (so) + sB_w + (h) * 256u, 0, W_AUX); \
        __builtin_amdgcn_raw_ptr_buffer_load_lds(rs, (LAS void*)(lds + 4 * HTB + SB(b, h) + ldsw + 8192), 16, voffB, (so) + sB_w + (h) * 256u + 32u * WROW, 0, W_AUX); } while (0)
#define LDA(dst, b, h) do { _Pragma("unroll") for (int m = 0; m < 4; ++m) { \
        dst[m][0] = *(const LAS s16x8*)(lds + SA(b, h) + aoff0 + m * 2048); \
        dst[m][1] = *(const LAS s16x8*)(lds + SA(b, h) + aoff1 + m * 2048); } } while (0)
#define TRR(dst, base, off) asm volatile("ds_read_b64_tr_b16 %0, %1 offset:%2" : "=v"(dst) : "v"(base), "i"(off) : "memory")
#define LDB(dst, b, h) do { _Pragma("unroll") for (int k = 0; k < 2; ++k) { \
        { s16x4 lo, hi; TRR(lo, boff0, SB(b, h) + k * 8192); TRR(hi, boff0, SB(b, h) + k * 8192 + 2048); dst[0][k] = __builtin_shufflevector(lo, hi, 0, 1, 2, 3, 4, 5, 6, 7); } \
        { s16x4 lo, hi; TRR(lo, boff1, SB(b, h) + k * 8192); TRR(hi, boff1, SB(b, h) + k * 8192 + 2048); dst[1][k] = __builtin_shufflevector(lo, hi, 0, 1, 2, 3, 4, 5, 6, 7); } } } while (0)
#define MF1(ai, bj, At, Bt, m, n, k) acc[ai][bj][m][n] = __builtin_amdgcn_mfma_f32_16x16x32_bf16(__builtin_bit_cast(bf16x8, Bt[n][k]), __builtin_bit_cast(bf16x8, At[m][k]), acc[ai][bj][m][n], 0, 0, 0)
#define MMA2(ai, At, Ba, Bb) do { __builtin_amdgcn_s_setprio(1); \
        _Pragma("unroll") for (int k = 0; k < 2; ++k) { \
            _Pragma("unroll") for (int m = 0; m < 4; ++m) _Pragma("unroll") for (int n = 0; n < 2; ++n) MF1(ai, 0, At, Ba, m, n, k); \
            _Pragma("unroll") for (int m = 0; m < 4; ++m) _Pragma("unroll") for (int n = 0; n < 2; ++n) MF1(ai, 1, At, Bb, m, n, k); } \
        __builtin_amdgcn_s_setprio(0); } while (0)
#define MMAK2(ai, At, Ba, Bb, k) do { \
            _Pragma("unroll") for (int m = 0; m < 4; ++m) _Pragma("unroll") for (int n = 0; n < 2; ++n) MF1(ai, 0, At, Ba, m, n, k); \
            _Pragma("unroll") for (int m = 0; m < 4; ++m) _Pragma("unroll") for (int n = 0; n < 2; ++n) MF1(ai, 1, At, Bb, m, n, k); } while (0)
#define MMA2H(ai, At, Ba, Bb, HOOK) do { __builtin_amdgcn_s_setprio(1); MMAK2(ai, At, Ba, Bb, 0); SCHED; HOOK; SCHED; MMAK2(ai, At, Ba, Bb, 1); __builtin_amdgcn_s_setprio(0); } while (0)
#define MMA2N(ai, At, Ba, Bb, SCALARS) do { __builtin_amdgcn_s_setprio(1); SCALARS; MMAK2(ai, At, Ba, Bb, 0); MMAK2(ai, At, Ba, Bb, 1); \
            _Pragma("unroll") for (int i_ = 0; i_ < 32; ++i_) { __builtin_amdgcn_sched_group_barrier(0x8, 1, 0); __builtin_amdgcn_sched_group_barrier(0x6, 1, 0); } \
            __builtin_amdgcn_s_setprio(0); } while (0)
#define WAIT_V(n) asm volatile("s_waitcnt vmcnt(" #n ")" ::: "memory")
#define WAIT_L(n) asm volatile("s_waitcnt lgkmcnt(" #n ")" ::: "memory")
#define BAR __builtin_amdgcn_s_barrier()
#define SCHED __builtin_amdgcn_sched_barrier(0)
#define PSTORE(reg, base) do { int l_; LANE_NOW(l_); asm volatile("global_store_dwordx4 %0, " reg ", %1 sc0 sc1\n\ts_nop 1" :: "v"(VOFFP(l_)), "s"(base) : "memory"); } while (0)
#define PLOAD(reg, c0, c1, c2, c3, base) do { int l_; LANE_NOW(l_); asm volatile("global_load_dwordx4 " reg ", %0, %1" :: "v"(VOFFP(l_)), "s"(base) : "memory", c0, c1, c2, c3); } while (0)
#define RTC(t) asm volatile("s_memrealtime %0\n\ts_waitcnt lgkmcnt(0)" : "=s"(t) :: "memory")

    f32x4 acc[2][2][4][2];
#pragma unroll
    for (int a = 0; a < 2; ++a)
#pragma unroll
        for (int b = 0; b < 2; ++b)
#pragma unroll
            for (int m = 0; m < 4; ++m)
#pragma unroll
                for (int n = 0; n < 2; ++n) acc[a][b][m][n] = (f32x4){0.f, 0.f, 0.f, 0.f};
    s16x8 At[4][2], B0a[2][2], B0b[2][2], B1[2][2];

    const auto rW = __builtin_amdgcn_make_buffer_rsrc((void*)p.w, 0, W_BYTES, 0x00020000);
    const char* const xown = uni(p.x + (size_t)rank * CHUNK_BYTES);
    auto rA = __builtin_amdgcn_make_buffer_rsrc((void*)xown, 0, CHUNK_BYTES, 0x00020000);
    const unsigned soB0 = (unsigned)(rank * KC) * WROW;

    STAGE_B(0, 0, rW, soB0); STAGE_A(0, 0, rA, 0u); STAGE_B(0, 1, rW, soB0);
    STAGE_B(1, 0, rW, soB0 + 64u * WROW); STAGE_A(0, 1, rA, 0u);
    WAIT_V(10);
    unsigned cntv;
    asm volatile("v_mov_b32 %0, v239" : "=v"(cntv) :: "memory");
    const unsigned call = (unsigned)__builtin_amdgcn_readfirstlane(cntv) / GRID + 1u;
    const size_t slot_off = WS_STAGE + (size_t)(call & 1u) * SLOT_BYTES;
    const char* const stage = uni(myws + slot_off);
    char* const pdst = uni(dws + slot_off + (size_t)rank * CHUNK_BYTES + (size_t)(pj * 16u) * AROW);
    char* const pflag = uni(dws + WS_FLAGS + (size_t)rank * 128 + pj * 4u);
    STAMP(1);
    if (pusher) PSTORE("v[240:243]", pdst);
    STAMP(38);
    STAGE_A(1, 0, rA, 128u); STAGE_B(1, 1, rW, soB0 + 64u * WROW);
    STAMP(39);
    if (pusher) WAIT_V(9); else WAIT_V(8);
    STAMP(47);
    BAR;
    LDB(B0a, 0, 0);
    if (wr == 1) BAR;
    STAMP(2);

    unsigned vAlo, vAhi, vSoB;
    {
        const int v = lane - 4;
        const int g = (v * 37) >> 8;
        const bool own = lane < 4;
        const int ch = own ? rank : ((rank + 1 + (v - g * 7)) & 7);
        const int sub = own ? lane : (g & 3);
        const unsigned long long ab = (unsigned long long)(own ? xown : stage + (size_t)ch * CHUNK_BYTES) + (unsigned)sub * 256u;
        vAlo = (unsigned)ab; vAhi = (unsigned)(ab >> 32) & 0xffffu;
        vSoB = (unsigned)(ch * KC + sub * 128) * WROW;
    }
    auto mkA = [&](int pr) { const unsigned lo = __builtin_amdgcn_readlane(vAlo, pr), hi = __builtin_amdgcn_readlane(vAhi, pr);
        return __builtin_amdgcn_make_buffer_rsrc((void*)(((unsigned long long)hi << 32) | lo), 0, CHUNK_BYTES, 0x00020000); };
    auto rA2 = mkA(1);
    unsigned soB2 = __builtin_amdgcn_readlane(vSoB, 1);

    const unsigned CHK = COMM_ON ? ((1u << 2) | (1u << 9) | (1u << 16) | (1u << 23)) : 0u;
    const unsigned PFM = 31u << 23;
    const unsigned PUSH = (3u << 1) | (3u << 9) | (3u << 16);
    const unsigned FLG = (1u << 1) | (1u << 3) | (1u << 11) | (1u << 18);
    const unsigned mA = __builtin_amdgcn_readfirstlane((pusher ? (PUSH | 1u) : (wid == 0 ? (CHK << 1) : 0u)) | (PREFETCH ? (PF_ROLL ? (wid != 0 ? PFM : 0u) : (1u << PF_U)) : 0u));
    const unsigned mA10 = __builtin_amdgcn_readfirstlane(pusher ? (PUSH << 1) : 0u);
    const unsigned mB = __builtin_amdgcn_readfirstlane(pusher ? (PUSH | FLG | 1u) : ((PREFETCH && !PF_ROLL && wid >= 2) ? (1u << PF_U) : 0u));
    const unsigned mC = __builtin_amdgcn_readfirstlane((pusher ? (PUSH | 1u) : (wid == 0 ? (CHK | ((COMM_ON && INV_MODE == 4) ? 2u : 0u)) : 0u)) | (1u << PF_U));
    const unsigned mD = __builtin_amdgcn_readfirstlane(pusher ? PUSH : 0u);
#define PFLAG(g) do { int l_; LANE_NOW(l_); if (l_ == 0) asm volatile("global_store_dword %0, %1, %2 sc0 sc1" :: "v"(0u), "v"(call), "s"(pflag + (g) * 1024) : "memory"); } while (0)
#define FLAG_VOFF(l) ((unsigned)(((rank + 1 + ((l) >> 3)) & 7) * 128 + ((l) & 7) * 16))
#define FLAG_LOAD "global_load_dwordx4 v[244:247], %0, %1 sc0 sc1"
#define FLAG_MIN(m) asm volatile("v_min3_u32 %0, v244, v245, v246\n\tv_min_u32 %0, %0, v247" : "=&v"(m) :: "memory")
#define PGRP (u < 9 ? 1 : (u < 16 ? 2 : 3))
#define POFF(j) ((unsigned)(2 * (u - (u < 9 ? 1 : (u < 16 ? 9 : 16))) + (j)) * 4u * AROW + (unsigned)PGRP * 256u)
#define GCHK ((u * 37) >> 8)
#define PREFETCH_W(pair, quarter) do { const unsigned so_ = __builtin_amdgcn_readlane(vSoB, pair) + (unsigned)(pm * 64 + (quarter) * 16) * WROW + (unsigned)pn * 512u; \
            int l_; LANE_NOW(l_); const unsigned vo_ = (unsigned)(l_ >> 2) * WROW + (unsigned)(l_ & 3) * 128u; \
            asm volatile("global_load_dword v249, %0, %1" :: "v"(vo_), "s"(p.w + so_) : "memory", "v249"); } while (0)
#define PREFETCH_A(pair0) do { int l_; LANE_NOW(l_); const int pr_ = min((pair0) + (l_ >> 5), 31); \
            const unsigned lo_ = __builtin_amdgcn_ds_bpermute(pr_ * 4, vAlo), hi_ = __builtin_amdgcn_ds_bpermute(pr_ * 4, vAhi); \
            const char* ad_ = (const char*)(((unsigned long long)hi_ << 32) | lo_) + (size_t)(pm * 256 + (jj >> 1) * 16 + ((l_ >> 1) & 15)) * AROW + (l_ & 1) * 128; \
            asm volatile("global_load_dword v250, %0, off" :: "v"(ad_) : "memory", "v250"); } while (0)
#define PREFETCH_ROLL do { int l_; LANE_NOW(l_); const int t0_ = 50 + 3 * (u - 23); \
            if (wid >= 2) { const int t_ = t0_ + ((wid - 2) >> 1); if (t_ < 64) { \
                const unsigned so_ = __builtin_amdgcn_readlane(vSoB, t_ >> 1) + (unsigned)((t_ & 1) * 64 + pm * 32 + (wid & 1) * 16) * WROW + (unsigned)pn * 512u; \
                const unsigned vo_ = (unsigned)(l_ >> 2) * WROW + (unsigned)(l_ & 3) * 128u; \
                asm volatile("global_load_dword v249, %0, %1" :: "v"(vo_), "s"(p.w + so_) : "memory", "v249"); } } \
            else { const int t_ = min(t0_ + (l_ >> 4), 63); \
                const unsigned lo_ = __builtin_amdgcn_ds_bpermute((t_ >> 1) * 4, vAlo), hi_ = __builtin_amdgcn_ds_bpermute((t_ >> 1) * 4, vAhi); \
                const char* ad_ = (const char*)(((unsigned long long)hi_ << 32) | lo_) + (size_t)(pm * 256 + (jj >> 1) * 16 + (l_ & 15)) * AROW + (t_ & 1) * 128; \
                asm volatile("global_load_dword v249, %0, off" :: "v"(ad_) : "memory", "v249"); } } while (0)
#define HK_A do { if (UNLIKELY(mA & bitu)) { if (PF_ROLL && u >= 23 && wid != 0) PREFETCH_ROLL; else if (!PF_ROLL && u == PF_U) PREFETCH_W(PF_U + 2 + (wid >> 2), wid & 3); else if (wid != 0) { if (u == 0) PSTORE("v[244:247]", pdst + 4 * AROW); else \
            PLOAD("v[240:243]", "v240", "v241", "v242", "v243", psrc + POFF(0)); } \
        else { \
            unsigned fm; int lc; LANE_NOW(lc); FLAG_MIN(fm); \
            if (!__all(lc >= 56 || fm >= call)) { \
                STAMPX_BEGIN; \
                unsigned long long t0, t1; RTC(t0); \
                for (;;) { \
                    unsigned tm; \
                    asm volatile(FLAG_LOAD "\n\tglobal_load_dword v248, %2, %3 sc1\n\ts_waitcnt vmcnt(0)" \
                                 :: "v"(FLAG_VOFF(lc)), "s"(myws + WS_FLAGS + GCHK * 1024), "v"(0u), "s"(myws + WS_TMO) : "memory", "v244", "v245", "v246", "v247", "v248"); \
                    FLAG_MIN(fm); \
                    if (__all(lc >= 56 || fm >= call)) break; \
                    asm volatile("v_mov_b32 %0, v248" : "=v"(tm) :: "memory"); \
                    RTC(t1); \
                    if ((unsigned)__builtin_amdgcn_readfirstlane(tm) == call || (unsigned)(t1 - t0) > (unsigned)DEADLINE_TICKS) { \
                        if (lc == 0) asm volatile("global_store_dword %0, %1, %2 sc1\n\ts_waitcnt vmcnt(0)" :: "v"(0u), "v"(call), "s"(myws + WS_TMO) : "memory"); \
                        break; } \
                    __builtin_amdgcn_s_sleep(2); \
                } \
                STAMPX_END(GCHK); \
            } } } } while (0)
#define HK_B do { if (UNLIKELY(mB & bitu)) { if (u == PF_U) { if (wid == 2) PREFETCH_A(30); else if (wid == 3) { if (PF_U < 28) PREFETCH_A(29); } else if (PF_U < 28) PREFETCH_W(31, wid & 3); } else if (u == 0) PSTORE("v[248:251]", pdst + 8 * AROW); else { \
            if (FLG & bitu) PFLAG(u == 1 ? 0 : PGRP); \
            if (PUSH & bitu) PLOAD("v[252:255]", "v252", "v253", "v254", "v255", psrc + POFF(1)); } } } while (0)
#define HK_C do { if (UNLIKELY(mC & bitu)) { \
            if (u == PF_U) asm volatile("global_load_dword v248, %0, %1 sc1" :: "v"(0u), "s"(myws + WS_TMO) : "memory", "v248"); \
            else if (wid != 0) { if (u == 0) PSTORE("v[252:255]", pdst + 12 * AROW); else PSTORE("v[240:243]", pdst + POFF(0)); } \
            else if (u == 1) asm volatile("buffer_inv sc0 sc1" ::: "memory"); \
            else { int lc; LANE_NOW(lc); asm volatile(FLAG_LOAD :: "v"(FLAG_VOFF(lc)), "s"(myws + WS_FLAGS + GCHK * 1024) : "memory", "v244", "v245", "v246", "v247"); } } } while (0)
#define HK_D do { if (UNLIKELY(mD & bitu)) PSTORE("v[252:255]", pdst + POFF(1)); } while (0)
#define PHASE(L1, L2, S1, S2, WAITV, MM) do { L1; L2; SCHED; S1; S2; WAITV; WAIT_L(0); BAR; SCHED; MM; BAR; SCHED; } while (0)
#define NEXT_PAIR do { rA = rA2; rA2 = mkA((u + 2) & 31); soB3 = __builtin_amdgcn_readlane(vSoB, (u + 2) & 31); } while (0)
    int u = 0;
    for (int seg = 0; seg < 5; ++seg) {
        const int nh = ((PF_ROLL ? 0x05334 : 0x12334) >> (4 * seg)) & 15, np = ((PF_ROLL ? 0x03445 : (PF_U == 28 ? 0x23445 : 0x32445)) >> (4 * seg)) & 15;
        for (int i = 0; i < nh; ++i, ++u) {
            if (u == 4) { STAMPC(44); STAMP(7); }
            const unsigned bitu = 1u << u;
            unsigned soB3;
            PHASE(LDA(At, 0, 0), LDB(B1, 0, 1), STAGE_B(0, 0, rW, soB2), STAGE_A(1, 1, rA, 128u),
                  do { WAIT_V(10); if (!UNLIKELY(mA10 & bitu)) WAIT_V(8); } while (0), MMA2H(0, At, B0a, B1, HK_A));
            PHASE(LDA(At, 0, 1), LDB(B0b, 1, 0), STAGE_A(0, 0, rA2, 0u), STAGE_B(0, 1, rW, soB2), WAIT_V(8), MMA2H(1, At, B0a, B1, HK_B));
            PHASE(LDA(At, 1, 0), LDB(B1, 1, 1), STAGE_B(1, 0, rW, soB2 + 64u * WROW), STAGE_A(0, 1, rA2, 0u), WAIT_V(8), MMA2H(0, At, B0b, B1, HK_C));
            PHASE(LDA(At, 1, 1), LDB(B0a, 0, 0), STAGE_A(1, 0, rA2, 128u), STAGE_B(1, 1, rW, soB2 + 64u * WROW), do { WAIT_V(8); NEXT_PAIR; } while (0),
                  MMA2H(1, At, B0b, B1, HK_D));
            soB2 = soB3;
        }
        for (int i = 0; i < np; ++i, ++u) {
            if (u == 4) { STAMPC(44); STAMP(7); }
            unsigned soB3;
            PHASE(LDA(At, 0, 0), LDB(B1, 0, 1), STAGE_B(0, 0, rW, soB2), STAGE_A(1, 1, rA, 128u), WAIT_V(8), MMA2(0, At, B0a, B1));
            PHASE(LDA(At, 0, 1), LDB(B0b, 1, 0), STAGE_A(0, 0, rA2, 0u), STAGE_B(0, 1, rW, soB2), WAIT_V(8), MMA2(1, At, B0a, B1));
            PHASE(LDA(At, 1, 0), LDB(B1, 1, 1), STAGE_B(1, 0, rW, soB2 + 64u * WROW), STAGE_A(0, 1, rA2, 0u), WAIT_V(8), MMA2(0, At, B0b, B1));
            PHASE(LDA(At, 1, 1), LDB(B0a, 0, 0), STAGE_A(1, 0, rA2, 128u), STAGE_B(1, 1, rW, soB2 + 64u * WROW), WAIT_V(8),
                  MMA2N(1, At, B0b, B1, NEXT_PAIR));
            soB2 = soB3;
        }
    }
    STAMP(34); STAMPC(46);

    int tz = threadIdx.x; asm volatile("" : "+v"(tz));
    const int l2 = tz & 63, fr2 = l2 & 15, fq2 = l2 >> 4;
    char* const obase = p.out + (size_t)(pm * 256 + wr * 64 + fr2) * WROW + (size_t)pn * 512 + wc * 64 + ((fq2 & 1) * 32 + (fq2 >> 1) * 16);
    bool bad;
#define EPI(ai) do { _Pragma("unroll") for (int m = 0; m < 4; ++m) { \
            char* rowp = obase + (size_t)((ai) * 128 + m * 16) * WROW; \
            _Pragma("unroll") for (int bj = 0; bj < 2; ++bj) { \
                const f32x4 v0 = acc[ai][bj][m][0], v1 = acc[ai][bj][m][1]; \
                unsigned a0 = relu_pk(cvt_pk_bf16(v0[0], v0[1])), a1 = relu_pk(cvt_pk_bf16(v0[2], v0[3])); \
                unsigned b0 = relu_pk(cvt_pk_bf16(v1[0], v1[1])), b1 = relu_pk(cvt_pk_bf16(v1[2], v1[3])); \
                auto r0 = __builtin_amdgcn_permlane16_swap(a0, b0, false, false); \
                auto r1 = __builtin_amdgcn_permlane16_swap(a1, b1, false, false); \
                u32x4 o; o.x = r0[0]; o.y = r1[0]; o.z = r0[1]; o.w = r1[1]; \
                if (bad) o = (u32x4){0x7fc07fc0u, 0x7fc07fc0u, 0x7fc07fc0u, 0x7fc07fc0u}; \
                asm volatile("global_store_dwordx4 %0, %1, off " OUT_ST "\n\ts_nop 1" :: "v"(rowp + bj * 256), "v"(o) : "memory"); } } } while (0)
    PHASE(LDA(At, 0, 0), LDB(B1, 0, 1), (void)0, STAGE_A(1, 1, rA, 128u), WAIT_V(6), MMA2(0, At, B0a, B1));
    { unsigned tm; asm volatile("v_mov_b32 %0, v248" : "=v"(tm) :: "memory"); bad = ((unsigned)__builtin_amdgcn_readfirstlane(tm) == call); }
    PHASE(LDA(At, 0, 1), LDB(B0b, 1, 0), (void)0, (void)0, WAIT_V(2), MMA2(1, At, B0a, B1));
    PHASE(LDA(At, 1, 0), LDB(B1, 1, 1), (void)0, (void)0, WAIT_V(0), MMA2(0, At, B0b, B1));
    LDA(At, 1, 1); SCHED;
    EPI(0);
    WAIT_L(0); BAR; SCHED;
    MMA2(1, At, B0b, B1);
    if (wr == 0) BAR;
    SCHED;
    STAMP(35);
    STAMPC(45);
    EPI(1);
    if (tid == 0) __hip_atomic_fetch_add((GAS unsigned*)(myws + WS_CNT), 1u, __ATOMIC_RELAXED, __HIP_MEMORY_SCOPE_AGENT);
}

extern "C" void kernel_launch(int rank, int nranks, ncclComm_t comm,
                              hipStream_t stream,
                              void* const* d_in, const int* in_sizes, int n_in,
                              void* d_out, int out_size,
                              const int* in_dtypes, int out_dtype,
                              void* const* d_out_all,
                              void* const* d_ws_all, size_t ws_bytes) {
    (void)comm; (void)in_dtypes; (void)out_dtype; (void)d_out_all;
    if (nranks != NR || n_in < 2 || rank < 0 || rank >= NR || ws_bytes < WS_NEED ||
        in_sizes[0] != KDIM * KC || in_sizes[1] != KDIM * NDIM || out_size != MLOC * NDIM) {
        fprintf(stderr, "kernel_launch: unexpected problem shape\n");
        return;
    }
    static bool first = true;
    if (first) {
        HIP_CHECK(hipFuncSetAttribute((const void*)a2a_gemm_kernel, hipFuncAttributeMaxDynamicSharedMemorySize, LDS_BYTES));
        first = false;
    }
    Params p;
    p.x = (const char*)d_in[0];
    p.w = (const char*)d_in[1];
    p.out = (char*)d_out;
    for (int i = 0; i < NR; ++i) p.ws[i] = (char*)d_ws_all[i];
    p.rank = rank;
    a2a_gemm_kernel<<<GRID, 512, LDS_BYTES, stream>>>(p);
    HIP_CHECK(hipGetLastError());
}
```

```cpp
#include <hip/hip_runtime.h>
#include <rccl/rccl.h>
#include <cstdio>
#include <cstdint>
#pragma clang diagnostic ignored "-Winline-asm"

#define HIP_CHECK(expr) do { hipError_t _e = (expr); if (_e != hipSuccess) { \
    fprintf(stderr, "HIP error %s at %s:%d\n", hipGetErrorString(_e), __FILE__, __LINE__); \
    return; } } while (0)

#define UNLIKELY(x) __builtin_expect(!!(x), 0)
#define LAS __attribute__((address_space(3)))
#define GAS __attribute__((address_space(1)))
typedef short s16x8 __attribute__((ext_vector_type(8)));
typedef short s16x4 __attribute__((ext_vector_type(4)));
typedef short s16x2 __attribute__((ext_vector_type(2)));
typedef __bf16 bf16x8 __attribute__((ext_vector_type(8)));
typedef float f32x4 __attribute__((ext_vector_type(4)));
typedef unsigned u32x4 __attribute__((ext_vector_type(4)));

constexpr int NR = 8;
constexpr int MLOC = 512;
constexpr int KDIM = 4096;
constexpr int NDIM = 32768;
constexpr int KC = KDIM / NR;
constexpr unsigned CHUNK_BYTES = MLOC * KC * 2;
constexpr int HTB = 16384;
constexpr int LDS_CNT = 8 * HTB;
constexpr int LDS_BYTES = 8 * HTB + 2048;
constexpr unsigned W_BYTES = (unsigned)KDIM * NDIM * 2u;
constexpr unsigned WROW = NDIM * 2;
constexpr unsigned AROW = KC * 2;
constexpr int GRID = 256;
constexpr size_t WS_CNT = 0;
constexpr size_t WS_TMO = 128;
constexpr size_t WS_FLAGS = 4096;
constexpr size_t WS_STAGE = 1u << 20;
constexpr size_t SLOT_BYTES = (size_t)NR * CHUNK_BYTES;
constexpr size_t WS_NEED = WS_STAGE + 2 * SLOT_BYTES;
constexpr unsigned DEADLINE_TICKS = 40000000u;

#define STAMP(i) do { } while (0)
#define STAMPC(i) do { } while (0)
#define STAMPX_BEGIN do { } while (0)
#define STAMPX_END(g) do { } while (0)
#define COMM_ON true
#define PREFETCH 1
#define USE_INV 0
#define OUT_ST "sc1 nt"
#define LANE_NOW(l) asm volatile("v_mbcnt_lo_u32_b32 %0, -1, 0\n\tv_mbcnt_hi_u32_b32 %0, -1, %0" : "=v"(l))
#define VOFFP(l) ((unsigned)((l) >> 4) * AROW + (unsigned)((l) & 15) * 16u)
#define FLAG_VOFF(l) ((unsigned)(((rank + 1 + ((l) >> 3)) & 7) * 128 + ((l) & 7) * 16))
#define PIECE_OFF(P) ((unsigned)((P) & 3) * 4u * AROW + (unsigned)((P) >> 2) * 256u)

struct Params {
    const char* x;
    const char* w;
    char* out;
    char* ws[NR];
    int rank;
};

__device__ __forceinline__ unsigned cvt_pk_bf16(float lo, float hi) {
    unsigned r; asm("v_cvt_pk_bf16_f32 %0, %1, %2" : "=v"(r) : "v"(lo), "v"(hi)); return r;
}
__device__ __forceinline__ unsigned relu_pk(unsigned v) {
    s16x2 a = __builtin_bit_cast(s16x2, v);
    s16x2 z = {0, 0};
    return __builtin_bit_cast(unsigned, __builtin_elementwise_max(a, z));
}
template <class T> __device__ __forceinline__ T* uni(T* ptr) {
    const unsigned long long v = (unsigned long long)ptr;
    const unsigned lo = __builtin_amdgcn_readfirstlane((unsigned)v), hi = __builtin_amdgcn_readfirstlane((unsigned)(v >> 32));
    return (T*)(((unsigned long long)hi << 32) | lo);
}
__device__ __forceinline__ u32x4 mk_desc(const void* base, unsigned bytes) {
    const unsigned long long v = (unsigned long long)base;
    return (u32x4){(unsigned)v, (unsigned)(v >> 32) & 0xffffu, bytes, 0x00020000u};
}

__global__ __launch_bounds__(512, 2) __attribute__((amdgpu_num_vgpr(120))) void a2a_gemm_kernel(Params p) {
    extern __shared__ __attribute__((aligned(16))) char smem_[];
    LAS char* lds = (LAS char*)smem_;
    const int tid = threadIdx.x;
    const int wid = __builtin_amdgcn_readfirstlane(tid >> 6);
    const int lane = tid & 63, wr = wid >> 2, wc = wid & 3, fr = lane & 15, fq = lane >> 4;
    const int bid = blockIdx.x;
    const int xcd = bid & 7, jj = bid >> 3;
    const int pm = jj & 1, pn = xcd * 16 + (jj >> 1);
    const int rank = p.rank;
    char* const myws = uni(p.ws[rank]);
    STAMP(0);

#define WAIT_V(n) asm volatile("s_waitcnt vmcnt(" #n ")" ::: "memory")
#define WAIT_L(n) asm volatile("s_waitcnt lgkmcnt(" #n ")" ::: "memory")
#define BAR __builtin_amdgcn_s_barrier()
#define SCHED __builtin_amdgcn_sched_barrier(0)
#define RTC(t) asm volatile("s_memrealtime %0\n\ts_waitcnt lgkmcnt(0)" : "=s"(t) :: "memory")

    {
        const auto rC = __builtin_amdgcn_make_buffer_rsrc((void*)myws, 0, 256, 0x00020000);
        __builtin_amdgcn_raw_ptr_buffer_load_lds(rC, (LAS void*)(lds + LDS_CNT + wid * 256), 4, 0, (int)WS_CNT, 0, 16);
    }
    const bool pusher = COMM_ON && (wid == 1) && (bid < 224);
    const unsigned dest = (unsigned)(rank + 1 + (bid >> 5)) & 7u;
    const unsigned pj = (unsigned)bid & 31u;
    const char* const psrc = uni(p.x + (size_t)(dest * MLOC + pj * 16u) * AROW);
    char* const dws = uni(p.ws[dest]);
    if (pusher) {
        const unsigned vp = VOFFP(lane);
        asm volatile("global_load_dwordx4 v[240:243], %0, %1" :: "v"(vp), "s"(psrc + PIECE_OFF(0)) : "memory", "v240", "v241", "v242", "v243");
        asm volatile("global_load_dwordx4 v[244:247], %0, %1" :: "v"(vp), "s"(psrc + PIECE_OFF(1)) : "memory", "v244", "v245", "v246", "v247");
        asm volatile("global_load_dwordx4 v[248:251], %0, %1" :: "v"(vp), "s"(psrc + PIECE_OFF(2)) : "memory", "v248", "v249", "v250", "v251");
        asm volatile("global_load_dwordx4 v[252:255], %0, %1" :: "v"(vp), "s"(psrc + PIECE_OFF(3)) : "memory", "v252", "v253", "v254", "v255");
    }

    const unsigned voffA = (unsigned)(lane >> 3) * AROW + (unsigned)(((lane & 7) ^ (lane >> 3)) * 16);
    const unsigned voffB = (unsigned)(((lane >> 3) & 1) * 8 + (lane >> 4)) * WROW + (unsigned)(((lane & 7) ^ ((lane >> 4) << 1)) * 16);
    const unsigned sA_w = (unsigned)(pm * 256 + wid * 8) * AROW;
    const unsigned sB_w = (unsigned)(((wid >> 2) & 1) * 16 + ((wid >> 1) & 1) * 4) * WROW + (unsigned)(wid & 1) * 128u + (unsigned)pn * 512u;
    const unsigned ldsw = (unsigned)wid * 1024u;
    const int arow = wr * 64 + fr;
    const unsigned aoff0 = (unsigned)arow * 128u + (unsigned)((fq ^ (fr & 7)) * 16);
    const unsigned aoff1 = aoff0 ^ 64u;
    const int q = (lane & 15) >> 2, pp = lane & 3;
    const unsigned bcom = (unsigned)(fq >> 1) * 4096u + (unsigned)(wc >> 1) * 1024u + (unsigned)q * 256u + (unsigned)(fq & 1) * 128u + (unsigned)pp * 8u;
    const unsigned ldsbase = (unsigned)(size_t)lds;
    const unsigned boff0 = ldsbase + 4 * HTB + bcom + (unsigned)(((2 * (wc & 1) + 0) ^ q) * 32);
    const unsigned boff1 = ldsbase + 4 * HTB + bcom + (unsigned)(((2 * (wc & 1) + 1) ^ q) * 32);

#define SA(b, h) (((b) * 2 + (h)) * HTB)
#define SB(b, h) (((b) * 2 + (h)) * HTB)
#define DMA_A(b, h, i, rs, so) __builtin_amdgcn_raw_ptr_buffer_load_lds(rs, (LAS void*)(lds + SA(b, h) + ldsw + (i) * 8192), 16, voffA, (so) + sA_w + (h) * 128u * AROW + (i) * 64u * AROW, 0, 0)
#define DMA_B(b, h, i, so) __builtin_amdgcn_raw_ptr_buffer_load_lds(rW, (LAS void*)(lds + 4 * HTB + SB(b, h) + ldsw + (i) * 8192), 16, voffB, (so) + sB_w + (h) * 256u + (i) * 32u * WROW, 0, 0)
#define STAGE_A(b, h, rs, so) do { DMA_A(b, h, 0, rs, so); DMA_A(b, h, 1, rs, so); } while (0)
#define STAGE_B(b, h, so) do { DMA_B(b, h, 0, so); DMA_B(b, h, 1, so); } while (0)
#define LDA_H(dst, b, h, hf) do { _Pragma("unroll") for (int m = 2 * (hf); m < 2 * (hf) + 2; ++m) { \
        dst[m][0] = *(const LAS s16x8*)(lds + SA(b, h) + aoff0 + m * 2048); \
        dst[m][1] = *(const LAS s16x8*)(lds + SA(b, h) + aoff1 + m * 2048); } } while (0)
#define TRR(dst, base, off) asm volatile("ds_read_b64_tr_b16 %0, %1 offset:%2" : "=v"(dst) : "v"(base), "i"(off) : "memory")
#define LDB_H(dst, b, h, k) do { \
        { s16x4 lo; s16x4 hi; TRR(lo, boff0, SB(b, h) + (k) * 8192); TRR(hi, boff0, SB(b, h) + (k) * 8192 + 2048); dst[0][k] = __builtin_shufflevector(lo, hi, 0, 1, 2, 3, 4, 5, 6, 7); } \
        { s16x4 lo; s16x4 hi; TRR(lo, boff1, SB(b, h) + (k) * 8192); TRR(hi, boff1, SB(b, h) + (k) * 8192 + 2048); dst[1][k] = __builtin_shufflevector(lo, hi, 0, 1, 2, 3, 4, 5, 6, 7); } } while (0)
#define LDA(dst, b, h) do { LDA_H(dst, b, h, 0); LDA_H(dst, b, h, 1); } while (0)
#define LDB(dst, b, h) do { LDB_H(dst, b, h, 0); LDB_H(dst, b, h, 1); } while (0)
#define MF1(ai, bj, At, Bt, m, n, k) acc[ai][bj][m][n] = __builtin_amdgcn_mfma_f32_16x16x32_bf16(__builtin_bit_cast(bf16x8, Bt[n][k]), __builtin_bit_cast(bf16x8, At[m][k]), acc[ai][bj][m][n], 0, 0, 0)
#define MMAK2(ai, At, Ba, Bb, k) do { \
            _Pragma("unroll") for (int m = 0; m < 4; ++m) _Pragma("unroll") for (int n = 0; n < 2; ++n) MF1(ai, 0, At, Ba, m, n, k); \
            _Pragma("unroll") for (int m = 0; m < 4; ++m) _Pragma("unroll") for (int n = 0; n < 2; ++n) MF1(ai, 1, At, Bb, m, n, k); } while (0)
#define MIX(n) do { _Pragma("unroll") for (int i_ = 0; i_ < (n); ++i_) { __builtin_amdgcn_sched_group_barrier(0x8, 1, 0); __builtin_amdgcn_sched_group_barrier(0x6, 1, 0); } } while (0)
#define MMA2(ai, At, Ba, Bb) do { __builtin_amdgcn_s_setprio(1); MMAK2(ai, At, Ba, Bb, 0); MMAK2(ai, At, Ba, Bb, 1); __builtin_amdgcn_s_setprio(0); } while (0)
#define MMA2N(ai, At, Ba, Bb, SCALARS) do { __builtin_amdgcn_s_setprio(1); SCALARS; MMAK2(ai, At, Ba, Bb, 0); MMAK2(ai, At, Ba, Bb, 1); MIX(32); __builtin_amdgcn_s_setprio(0); } while (0)
#define MMA2H(ai, At, Ba, Bb, SCALARS, HOOK) do { __builtin_amdgcn_s_setprio(1); SCALARS; MMAK2(ai, At, Ba, Bb, 0); MIX(16); SCHED; HOOK; SCHED; MMAK2(ai, At, Ba, Bb, 1); __builtin_amdgcn_s_setprio(0); } while (0)

    f32x4 acc[2][2][4][2];
#pragma unroll
    for (int a = 0; a < 2; ++a)
#pragma unroll
        for (int b = 0; b < 2; ++b)
#pragma unroll
            for (int m = 0; m < 4; ++m)
#pragma unroll
                for (int n = 0; n < 2; ++n) acc[a][b][m][n] = (f32x4){0.f, 0.f, 0.f, 0.f};
    s16x8 At[4][2], B0a[2][2], B0b[2][2], B1[2][2];

    const auto rW = __builtin_amdgcn_make_buffer_rsrc((void*)p.w, 0, W_BYTES, 0x00020000);
    const char* const xown = uni(p.x + (size_t)rank * CHUNK_BYTES);
    auto rA = __builtin_amdgcn_make_buffer_rsrc((void*)xown, 0, CHUNK_BYTES, 0x00020000);
    const unsigned soB0 = (unsigned)(rank * KC) * WROW;

    STAGE_B(0, 0, soB0); STAGE_A(0, 0, rA, 0u); STAGE_B(0, 1, soB0);
    STAGE_B(1, 0, soB0 + 64u * WROW); STAGE_A(0, 1, rA, 0u);
    WAIT_V(10);
    unsigned cntv;
    asm volatile("ds_read_b32 %0, %1\n\ts_waitcnt lgkmcnt(0)" : "=&v"(cntv) : "v"(ldsbase + LDS_CNT + (unsigned)wid * 256u) : "memory");
    const unsigned call = (unsigned)__builtin_amdgcn_readfirstlane(cntv) / GRID + 1u;
    const size_t slot_off = WS_STAGE + (size_t)(call & 1u) * SLOT_BYTES;
    const char* const stage = uni(myws + slot_off);
    char* const pdst = uni(dws + slot_off + (size_t)rank * CHUNK_BYTES + (size_t)(pj * 16u) * AROW);
    char* const pflag = uni(dws + WS_FLAGS + (size_t)rank * 128 + pj * 4u);
    STAMP(1);
    if (pusher) {
        const unsigned vp = VOFFP(lane);
        asm volatile("global_store_dwordx4 %0, v[240:243], %1 sc0 sc1\n\tglobal_store_dwordx4 %0, v[244:247], %2 sc0 sc1\n\ts_nop 1"
                     :: "v"(vp), "s"(pdst + PIECE_OFF(0)), "s"(pdst + PIECE_OFF(1)) : "memory");
    }
    STAGE_A(1, 0, rA, 128u); STAGE_B(1, 1, soB0 + 64u * WROW);
    if (pusher) WAIT_V(10); else WAIT_V(8);
    BAR;
    LDB(B0a, 0, 0);
    if (wr == 1) BAR;
    STAMP(2);

    unsigned vSoB;
    {
        const int v = lane - 4;
        const int g = (v * 37) >> 8;
        const bool own = lane < 4;
        const int ch = own ? rank : ((rank + 1 + (v - g * 7)) & 7);
        const int sub = own ? lane : (g & 3);
        vSoB = (unsigned)(ch * KC + sub * 128) * WROW;
    }
    auto mkA = [&](int pr, unsigned so) {
        const char* const base = (pr < 4 ? xown : stage + (size_t)(so >> 25) * CHUNK_BYTES) + ((so >> 23) & 3u) * 256u;
        return __builtin_amdgcn_make_buffer_rsrc((void*)base, 0, CHUNK_BYTES, 0x00020000); };
    unsigned soB2 = __builtin_amdgcn_readlane(vSoB, 1);
    auto rA2 = mkA(1, soB2);

    unsigned vPo, vH;
    {
        const unsigned po = PIECE_OFF(min(lane, 15));
        vPo = pusher ? po : (lane == 9 ? (unsigned)WS_FLAGS : (lane == 23 ? (unsigned)WS_FLAGS + 1024u : 0u));
        vH = pusher ? VOFFP(lane) : FLAG_VOFF(lane);
    }
    const unsigned vcall = call;
    const u32x4 dL = pusher ? mk_desc(psrc, 16384u) : mk_desc(myws, (COMM_ON && wid == 0) ? (unsigned)WS_FLAGS + 2048u : 0u);
    const unsigned mS = __builtin_amdgcn_readfirstlane(pusher ? 0x7Fu : 0u);
    const unsigned mF = __builtin_amdgcn_readfirstlane(pusher ? 0x82u : 0u);
    const unsigned mR = __builtin_amdgcn_readfirstlane((COMM_ON && wid == 0) ? ((USE_INV ? 2u : 0u) | (1u << 3) | (1u << 10)) : 0u);
    const unsigned m10 = __builtin_amdgcn_readfirstlane(0x4FEu | (pusher ? 1u : 0u));
#define SLOT(reg, c0, c1, c2, c3, dS, oS, oL) asm volatile("buffer_store_dwordx4 " reg ", %0, %1, %2 offen sc0 sc1\n\ts_nop 1\n\tbuffer_load_dwordx4 " reg ", %0, %3, %4 offen sc0 sc1" \
            :: "v"(vH), "s"(dS), "s"(oS), "s"(dL), "s"(oL) : "memory", c0, c1, c2, c3)
#define FLAG_MIN(m) asm volatile("v_min3_u32 %0, v252, v253, v254\n\tv_min_u32 %0, %0, v255" : "=&v"(m) :: "memory")
#define PREFETCH_W(pair, quarter) do { const unsigned so_ = __builtin_amdgcn_readlane(vSoB, pair) + (unsigned)(pm * 64 + (quarter) * 16) * WROW + (unsigned)pn * 512u; \
            int l_; LANE_NOW(l_); const unsigned vo_ = (unsigned)(l_ >> 2) * WROW + (unsigned)(l_ & 3) * 128u; \
            asm volatile("global_load_dword v249, %0, %1" :: "v"(vo_), "s"(p.w + so_) : "memory", "v249"); } while (0)
#define PREFETCH_A(pair0) do { int l_; LANE_NOW(l_); const int pr_ = min((pair0) + (l_ >> 5), 31); \
            const unsigned so_ = __builtin_amdgcn_ds_bpermute(pr_ * 4, vSoB); \
            const char* ad_ = stage + (size_t)((so_ >> 25) * CHUNK_BYTES + ((so_ >> 23) & 3u) * 256u) + (size_t)(pm * 256 + (jj >> 1) * 16 + ((l_ >> 1) & 15)) * AROW + (l_ & 1) * 128; \
            asm volatile("global_load_dword v250, %0, off" :: "v"(ad_) : "memory", "v250"); } while (0)
#define HK_RARE do { if (UNLIKELY(mR & bitu)) { \
        if (u == 1) asm volatile("buffer_inv sc0 sc1" ::: "memory"); \
        else { \
            unsigned fm; int lc; LANE_NOW(lc); FLAG_MIN(fm); \
            if (!__all(lc >= 56 || fm >= call)) { \
                STAMPX_BEGIN; \
                unsigned long long t0; unsigned long long t1; RTC(t0); \
                for (;;) { \
                    unsigned tm; \
                    asm volatile("global_load_dwordx4 v[252:255], %0, %1 sc0 sc1\n\tglobal_load_dword v244, %2, %3 sc1\n\ts_waitcnt vmcnt(0)" \
                                 :: "v"(vH), "s"(myws + WS_FLAGS + (u < 9 ? 0 : 1024)), "v"(0u), "s"(myws + WS_TMO) : "memory", "v252", "v253", "v254", "v255", "v244"); \
                    FLAG_MIN(fm); \
                    if (__all(lc >= 56 || fm >= call)) break; \
                    asm volatile("v_mov_b32 %0, v244" : "=v"(tm) :: "memory"); \
                    RTC(t1); \
                    if ((unsigned)__builtin_amdgcn_readfirstlane(tm) == call || (unsigned)(t1 - t0) > DEADLINE_TICKS) { \
                        if (lc == 0) asm volatile("global_store_dword %0, %1, %2 sc1\n\ts_waitcnt vmcnt(0)" :: "v"(0u), "v"(call), "s"(myws + WS_TMO) : "memory"); \
                        break; } \
                    __builtin_amdgcn_s_sleep(2); \
                } \
                STAMPX_END(u < 9 ? 0 : 1); \
            } } } } while (0)
#define PHASE(LA, LB, ab, ah, bb, bh, D0, D1, D2, D3, WAITV, MM) do { \
            LDA_H(LA, ab, ah, 0); SCHED; D0; SCHED; LDA_H(LA, ab, ah, 1); SCHED; D1; SCHED; LDB_H(LB, bb, bh, 0); SCHED; D2; SCHED; LDB_H(LB, bb, bh, 1); SCHED; D3; \
            WAITV; WAIT_L(0); BAR; SCHED; MM; BAR; SCHED; } while (0)
#define NEXT_PAIR do { rA = rA2; soB3 = __builtin_amdgcn_readlane(vSoB, (u + 2) & 31); rA2 = mkA(u + 2, soB3); } while (0)
#define PAIR_BODY(WA, WB, WC, WD, MA, MB, MC) do { unsigned soB3; \
            PHASE(At, B1, 0, 0, 0, 1, DMA_B(0, 0, 0, soB2), DMA_B(0, 0, 1, soB2), DMA_A(1, 1, 0, rA, 128u), DMA_A(1, 1, 1, rA, 128u), WA, MA); \
            PHASE(At, B0b, 0, 1, 1, 0, DMA_A(0, 0, 0, rA2, 0u), DMA_A(0, 0, 1, rA2, 0u), DMA_B(0, 1, 0, soB2), DMA_B(0, 1, 1, soB2), WB, MB); \
            PHASE(At, B1, 1, 0, 1, 1, DMA_B(1, 0, 0, soB2 + 64u * WROW), DMA_B(1, 0, 1, soB2 + 64u * WROW), DMA_A(0, 1, 0, rA2, 0u), DMA_A(0, 1, 1, rA2, 0u), WC, MC); \
            PHASE(At, B0a, 1, 1, 0, 0, DMA_A(1, 0, 0, rA2, 128u), DMA_A(1, 0, 1, rA2, 128u), DMA_B(1, 1, 0, soB2 + 64u * WROW), DMA_B(1, 1, 1, soB2 + 64u * WROW), WD, \
                  MMA2N(1, At, B0b, B1, NEXT_PAIR)); \
            soB2 = soB3; } while (0)
#define ITSTAMP do { } while (0)
    int u = 0;
    for (int seg = 0; seg < 3; ++seg) {
        const int nh = (0x000208 >> (8 * seg)) & 255, np = (0x031001 >> (8 * seg)) & 255;
        if (seg == 2) {
            ITSTAMP;
            PAIR_BODY(WAIT_V(8), WAIT_V(10), WAIT_V(11), WAIT_V(9),
                      MMA2H(0, At, B0a, B1, (void)0, do { asm volatile("global_load_dword v244, %0, %1 sc1" :: "v"(0u), "s"(myws + WS_TMO) : "memory", "v244"); PREFETCH_W(29 + (wid >> 2), wid & 3); } while (0)),
                      MMA2H(1, At, B0a, B1, (void)0, do { if (wid == 4) PREFETCH_A(29); else if (wid == 5) PREFETCH_A(31); else PREFETCH_W(31, wid & 3); } while (0)),
                      MMA2(0, At, B0b, B1));
            ++u;
        }
        for (int i = 0; i < nh; ++i, ++u) {
            ITSTAMP;
            const unsigned bitu = 1u << u;
            u32x4 dS, dF; unsigned oS, oL, oF;
            PAIR_BODY(do { WAIT_V(10); if (UNLIKELY(!(m10 & bitu))) WAIT_V(8); } while (0), WAIT_V(10), WAIT_V(12), WAIT_V(12),
                      MMA2H(0, At, B0a, B1,
                            do { dS = mk_desc(pdst, (mS & bitu) ? 16384u : 0u); oS = __builtin_amdgcn_readlane(vPo, 2 * u + 2); oL = __builtin_amdgcn_readlane(vPo, 2 * u + 4); } while (0),
                            do { SLOT("v[248:251]", "v248", "v249", "v250", "v251", dS, oS, oL); HK_RARE; } while (0)),
                      MMA2H(1, At, B0a, B1,
                            do { dF = mk_desc(pflag, (mF & bitu) ? 2048u : 0u); oF = (u == 7) ? 1024u : 0u; } while (0),
                            asm volatile("buffer_store_dword %0, off, %1, %2 sc0 sc1\n\tbuffer_store_dword %0, off, %1, %2 sc0 sc1" :: "v"(vcall), "s"(dF), "s"(oF) : "memory")),
                      MMA2H(0, At, B0b, B1,
                            do { oS = __builtin_amdgcn_readlane(vPo, 2 * u + 3); oL = __builtin_amdgcn_readlane(vPo, 2 * u + 5); } while (0),
                            SLOT("v[252:255]", "v252", "v253", "v254", "v255", dS, oS, oL)));
        }
        if (seg == 1) { STAMPC(44); STAMP(7); }
        for (int i = 0; i < np; ++i, ++u) {
            ITSTAMP;
            PAIR_BODY(WAIT_V(8), WAIT_V(8), WAIT_V(8), WAIT_V(8), MMA2(0, At, B0a, B1), MMA2(1, At, B0a, B1), MMA2(0, At, B0b, B1));
        }
        if (seg == 1) { STAMPC(46); STAMP(33); }
    }
    STAMP(34);

    int tz = threadIdx.x; asm volatile("" : "+v"(tz));
    const int l2 = tz & 63, fr2 = l2 & 15, fq2 = l2 >> 4;
    char* const obase = p.out + (size_t)(pm * 256 + wr * 64 + fr2) * WROW + (size_t)pn * 512 + wc * 64 + ((fq2 & 1) * 32 + (fq2 >> 1) * 16);
    bool bad;
#define EPI(ai) do { _Pragma("unroll") for (int m = 0; m < 4; ++m) { \
            char* rowp = obase + (size_t)((ai) * 128 + m * 16) * WROW; \
            _Pragma("unroll") for (int bj = 0; bj < 2; ++bj) { \
                const f32x4 v0 = acc[ai][bj][m][0], v1 = acc[ai][bj][m][1]; \
                unsigned a0 = relu_pk(cvt_pk_bf16(v0[0], v0[1])), a1 = relu_pk(cvt_pk_bf16(v0[2], v0[3])); \
                unsigned b0 = relu_pk(cvt_pk_bf16(v1[0], v1[1])), b1 = relu_pk(cvt_pk_bf16(v1[2], v1[3])); \
                auto r0 = __builtin_amdgcn_permlane16_swap(a0, b0, false, false); \
                auto r1 = __builtin_amdgcn_permlane16_swap(a1, b1, false, false); \
                u32x4 o; o.x = r0[0]; o.y = r1[0]; o.z = r0[1]; o.w = r1[1]; \
                if (bad) o = (u32x4){0x7fc07fc0u, 0x7fc07fc0u, 0x7fc07fc0u, 0x7fc07fc0u}; \
                asm volatile("global_store_dwordx4 %0, %1, off " OUT_ST "\n\ts_nop 1" :: "v"(rowp + bj * 256), "v"(o) : "memory"); } } } while (0)
#define PHASE_L(L1, L2, S2, WAITV, MM) do { L1; L2; SCHED; S2; WAITV; WAIT_L(0); BAR; SCHED; MM; BAR; SCHED; } while (0)
    PHASE_L(LDA(At, 0, 0), LDB(B1, 0, 1), STAGE_A(1, 1, rA, 128u), WAIT_V(6), MMA2(0, At, B0a, B1));
    { unsigned tm; asm volatile("v_mov_b32 %0, v244" : "=v"(tm) :: "memory"); bad = ((unsigned)__builtin_amdgcn_readfirstlane(tm) == call); }
    PHASE_L(LDA(At, 0, 1), LDB(B0b, 1, 0), (void)0, WAIT_V(2), MMA2(1, At, B0a, B1));
    PHASE_L(LDA(At, 1, 0), LDB(B1, 1, 1), (void)0, WAIT_V(0), MMA2(0, At, B0b, B1));
    LDA(At, 1, 1); SCHED;
    EPI(0);
    WAIT_L(0); BAR; SCHED;
    MMA2(1, At, B0b, B1);
    if (wr == 0) BAR;
    SCHED;
    STAMP(35);
    EPI(1);
    if (tid == 0) __hip_atomic_fetch_add((GAS unsigned*)(myws + WS_CNT), 1u, __ATOMIC_RELAXED, __HIP_MEMORY_SCOPE_AGENT);
}

extern "C" void kernel_launch(int rank, int nranks, ncclComm_t comm,
                              hipStream_t stream,
                              void* const* d_in, const int* in_sizes, int n_in,
                              void* d_out, int out_size,
                              const int* in_dtypes, int out_dtype,
                              void* const* d_out_all,
                              void* const* d_ws_all, size_t ws_bytes) {
    (void)comm; (void)in_dtypes; (void)out_dtype; (void)d_out_all;
    if (nranks != NR || n_in < 2 || rank < 0 || rank >= NR || ws_bytes < WS_NEED ||
        in_sizes[0] != KDIM * KC || in_sizes[1] != KDIM * NDIM || out_size != MLOC * NDIM) {
        fprintf(stderr, "kernel_launch: unexpected problem shape\n");
        return;
    }
    static bool first = true;
    if (first) {
        HIP_CHECK(hipFuncSetAttribute((const void*)a2a_gemm_kernel, hipFuncAttributeMaxDynamicSharedMemorySize, LDS_BYTES));
        first = false;
    }
    Params p;
    p.x = (const char*)d_in[0];
    p.w = (const char*)d_in[1];
    p.out = (char*)d_out;
    for (int i = 0; i < NR; ++i) p.ws[i] = (char*)d_ws_all[i];
    p.rank = rank;
    a2a_gemm_kernel<<<GRID, 512, LDS_BYTES, stream>>>(p);
    HIP_CHECK(hipGetLastError());
}
```

```cpp
#include <hip/hip_runtime.h>
#include <rccl/rccl.h>
#include <cstdio>
#include <cstdint>
#pragma clang diagnostic ignored "-Winline-asm"

#define HIP_CHECK(expr) do { hipError_t _e = (expr); if (_e != hipSuccess) { \
    fprintf(stderr, "HIP error %s at %s:%d\n", hipGetErrorString(_e), __FILE__, __LINE__); \
    return; } } while (0)

#define UNLIKELY(x) __builtin_expect(!!(x), 0)
#define LAS __attribute__((address_space(3)))
#define GAS __attribute__((address_space(1)))
typedef short s16x2 __attribute__((ext_vector_type(2)));
typedef unsigned u32x4 __attribute__((ext_vector_type(4)));

constexpr int NR = 8;
constexpr int MLOC = 512;
constexpr int KDIM = 4096;
constexpr int NDIM = 32768;
constexpr int KC = KDIM / NR;
constexpr unsigned CHUNK_BYTES = MLOC * KC * 2;
constexpr int HTB = 16384;
constexpr int LDS_CNT = 8 * HTB;
constexpr int LDS_TAB = 8 * HTB + 2048;
constexpr int LDS_BYTES = 8 * HTB + 2048 + 256;
constexpr unsigned W_BYTES = (unsigned)KDIM * NDIM * 2u;
constexpr unsigned WROW = NDIM * 2;
constexpr unsigned AROW = KC * 2;
constexpr int GRID = 256;
constexpr size_t WS_CNT = 0;
constexpr size_t WS_TMO = 128;
constexpr size_t WS_FLAGS = 4096;
constexpr size_t WS_STAGE = 1u << 20;
constexpr size_t SLOT_BYTES = (size_t)NR * CHUNK_BYTES;
constexpr size_t WS_NEED = WS_STAGE + 2 * SLOT_BYTES;
constexpr unsigned DEADLINE_TICKS = 40000000u;
constexpr unsigned PATIENCE_TICKS = 100u;

#define LANE_NOW(l) asm volatile("v_mbcnt_lo_u32_b32 %0, -1, 0\n\tv_mbcnt_hi_u32_b32 %0, -1, %0" : "=v"(l))
#define VOFFP(l) ((unsigned)((l) >> 4) * AROW + (unsigned)((l) & 15) * 16u)
#define CNT_VOFF(l) ((unsigned)(((rank + 1 + ((l) & 7)) & 7) * 128 + (((l) >> 3) & 3) * 32 + ((l) >> 5) * 16))
#define PIECE_OFF(P) ((unsigned)((P) & 3) * 4u * AROW + (unsigned)((P) >> 2) * 256u)

template <bool V> struct BoolTag { static constexpr bool value = V; };
struct Params {
    const char* x;
    const char* w;
    char* out;
    char* ws[NR];
    int rank;
};

__device__ __forceinline__ unsigned relu_pk(unsigned v) {
    s16x2 a = __builtin_bit_cast(s16x2, v);
    s16x2 z = {0, 0};
    return __builtin_bit_cast(unsigned, __builtin_elementwise_max(a, z));
}
template <class T> __device__ __forceinline__ T* uni(T* ptr) {
    const unsigned long long v = (unsigned long long)ptr;
    const unsigned lo = __builtin_amdgcn_readfirstlane((unsigned)v), hi = __builtin_amdgcn_readfirstlane((unsigned)(v >> 32));
    return (T*)(((unsigned long long)hi << 32) | lo);
}
__device__ __forceinline__ u32x4 mk_desc(const void* base, unsigned bytes) {
    const unsigned long long v = (unsigned long long)base;
    return (u32x4){(unsigned)__builtin_amdgcn_readfirstlane((unsigned)v), (unsigned)__builtin_amdgcn_readfirstlane((unsigned)(v >> 32) & 0xffffu),
                   (unsigned)__builtin_amdgcn_readfirstlane(bytes), 0x00020000u};
}

__global__ __launch_bounds__(512, 2) __attribute__((amdgpu_num_vgpr(18))) void a2a_gemm_kernel(Params p) {
    extern __shared__ __attribute__((aligned(16))) char smem_[];
    LAS char* lds = (LAS char*)smem_;
    const int tid = threadIdx.x;
    const int wid = __builtin_amdgcn_readfirstlane(tid >> 6);
    const int lane = tid & 63, wr = wid >> 2, wc = wid & 3, fr = lane & 15, fq = lane >> 4;
    const int bid = blockIdx.x;
    const int xcd = bid & 7, jj = bid >> 3;
    const int pm = jj & 1, pn = xcd * 16 + (jj >> 1);
    const int rank = p.rank;
    char* const myws = uni(p.ws[rank]);

#define WAIT_V(n) asm volatile("s_waitcnt vmcnt(" #n ")" ::: "memory")
#define WAIT_L(n) asm volatile("s_waitcnt lgkmcnt(" #n ")" ::: "memory")
#define BAR __builtin_amdgcn_s_barrier()
#define SCHED __builtin_amdgcn_sched_barrier(0)
#define RTC(t) asm volatile("s_memrealtime %0\n\ts_waitcnt lgkmcnt(0)" : "=s"(t) :: "memory")

    asm volatile("s_mov_b32 m0, %1\n\ts_nop 0\n\tbuffer_load_dword off, %0, 0 sc1 lds"
                 :: "s"(mk_desc(myws + WS_CNT, 4u)), "s"((unsigned)(size_t)lds + LDS_CNT + (unsigned)wid * 256u) : "memory");
    const int pe = (bid >> 3) * 4 + ((bid & 7) >> 1);
    const bool pwg = !(bid & 1) && pe < 112;
    const bool pusher = pwg && ((wid & 3) == 1);
    const bool pusher0 = pwg && ((wid & 3) == 2);
    const unsigned punit = (unsigned)(2 * pe + (wid >> 2));
    const unsigned dest = (unsigned)(rank + 1 + (punit >> 5)) & 7u;
    const unsigned pj = punit & 31u;
    const char* const psrc = uni(p.x + (size_t)(dest * MLOC + pj * 16u) * AROW);
    char* const dws = uni(p.ws[dest]);
    if (pusher || pusher0) {
        const unsigned vp = VOFFP(lane);
        const char* const s0 = psrc + (pusher ? PIECE_OFF(2) : PIECE_OFF(0));
        asm volatile("global_load_dwordx4 v[248:251], %0, %1" :: "v"(vp), "s"(s0) : "memory", "v248", "v249", "v250", "v251");
        asm volatile("global_load_dwordx4 v[252:255], %0, %1" :: "v"(vp), "s"(s0 + PIECE_OFF(1)) : "memory", "v252", "v253", "v254", "v255");
    }

    const unsigned voffA = (unsigned)(lane >> 3) * AROW + (unsigned)(((lane & 7) ^ (lane >> 3)) * 16);
    const unsigned voffB = (unsigned)(((lane >> 3) & 1) * 8 + (lane >> 4)) * WROW + (unsigned)(((lane & 7) ^ ((lane >> 4) << 1)) * 16);
    const unsigned sA_w = (unsigned)(pm * 256 + wid * 8) * AROW;
    const unsigned sB_w = (unsigned)(((wid >> 2) & 1) * 16 + ((wid >> 1) & 1) * 4) * WROW + (unsigned)(wid & 1) * 128u + (unsigned)pn * 512u;
    const unsigned ldsw = (unsigned)(size_t)lds + (unsigned)wid * 1024u;
    const int arow = wr * 64 + fr;
    const unsigned aoff0 = (unsigned)(size_t)lds + (unsigned)arow * 128u + (unsigned)((fq ^ (fr & 7)) * 16);
    const unsigned aoff1 = aoff0 ^ 64u;
    const int q = (lane & 15) >> 2, pp = lane & 3;
    const unsigned bcom = (unsigned)(fq >> 1) * 4096u + (unsigned)(wc >> 1) * 1024u + (unsigned)q * 256u + (unsigned)(fq & 1) * 128u + (unsigned)pp * 8u;
    const unsigned ldsbase = (unsigned)(size_t)lds;
    const unsigned boff0 = ldsbase + 4 * HTB + bcom + (unsigned)(((2 * (wc & 1) + 0) ^ q) * 32);
    const unsigned boff1 = ldsbase + 4 * HTB + bcom + (unsigned)(((2 * (wc & 1) + 1) ^ q) * 32);

#define SA(b, h) (((b) * 2 + (h)) * HTB)
#define SB(b, h) (((b) * 2 + (h)) * HTB)
#define DMA(voff, desc, soff, ldsoff, pol) asm volatile("s_add_u32 m0, %3, %4\n\ts_nop 0\n\tbuffer_load_dwordx4 %0, %1, %2 offen " pol "lds" \
            :: "v"(voff), "s"(desc), "s"(soff), "s"(ldsw), "i"(ldsoff) : "memory", "scc")
#define DMA_A(b, h, i, rs, so) DMA(voffA, rs, (so) + (h) * 128u * AROW + (i) * 64u * AROW, SA(b, h) + (i) * 8192, "")
#define DMA_B(b, h, i, so) do { if constexpr (STREAM_W) DMA(voffB, rW, (so) + (h) * 256u + (i) * 32u * WROW, 4 * HTB + SB(b, h) + (i) * 8192, "sc0 sc1 nt "); \
                                else DMA(voffB, rW, (so) + (h) * 256u + (i) * 32u * WROW, 4 * HTB + SB(b, h) + (i) * 8192, ""); } while (0)
#define STAGE_A(b, h, rs, so) do { DMA_A(b, h, 0, rs, so); DMA_A(b, h, 1, rs, so); } while (0)
#define STAGE_B(b, h, so) do { DMA_B(b, h, 0, so); DMA_B(b, h, 1, so); } while (0)
#define ACC(ai, bj, m, n) (36 + ((((ai) * 2 + (bj)) * 4 + (m)) * 2 + (n)) * 4)
#define ATR(m, k) (164 + ((m) * 2 + (k)) * 4)
#define B0a 196
#define B0b 212
#define B1 228
#define BR(base, n, k) ((base) + ((n) * 2 + (k)) * 4)
#define DSR128(reg, addr, off) asm volatile("ds_read_b128 v[%c0:%c1], %2 offset:%c3" :: "i"(reg), "i"((reg) + 3), "v"(addr), "i"(off) : "memory")
#define TRR(reg, addr, off) asm volatile("ds_read_b64_tr_b16 v[%c0:%c1], %2 offset:%c3" :: "i"(reg), "i"((reg) + 1), "v"(addr), "i"(off) : "memory")
#define LDA_H(b, h, hf) do { _Pragma("unroll") for (int m = 2 * (hf); m < 2 * (hf) + 2; ++m) { \
        DSR128(ATR(m, 0), aoff0, SA(b, h) + m * 2048); DSR128(ATR(m, 1), aoff1, SA(b, h) + m * 2048); } } while (0)
#define LDB_H(dst, b, h, k) do { \
        TRR(BR(dst, 0, k), boff0, SB(b, h) + (k) * 8192); TRR(BR(dst, 0, k) + 2, boff0, SB(b, h) + (k) * 8192 + 2048); \
        TRR(BR(dst, 1, k), boff1, SB(b, h) + (k) * 8192); TRR(BR(dst, 1, k) + 2, boff1, SB(b, h) + (k) * 8192 + 2048); } while (0)
#define LDA(b, h) do { LDA_H(b, h, 0); LDA_H(b, h, 1); } while (0)
#define LDB(dst, b, h) do { LDB_H(dst, b, h, 0); LDB_H(dst, b, h, 1); } while (0)
#define MF1(ai, bj, Bt, m, n, k) asm volatile("v_mfma_f32_16x16x32_bf16 v[%c0:%c1], v[%c2:%c3], v[%c4:%c5], v[%c0:%c1]" \
        :: "i"(ACC(ai, bj, m, n)), "i"(ACC(ai, bj, m, n) + 3), "i"(BR(Bt, n, k)), "i"(BR(Bt, n, k) + 3), "i"(ATR(m, k)), "i"(ATR(m, k) + 3))
#define MMAQ(ai, bj, Bt, k, m0_) do { MF1(ai, bj, Bt, m0_, 0, k); MF1(ai, bj, Bt, m0_, 1, k); MF1(ai, bj, Bt, m0_ + 1, 0, k); MF1(ai, bj, Bt, m0_ + 1, 1, k); } while (0)
#define MMAK2S(ai, Ba, Bb, k, S0, S1, S2) do { MMAQ(ai, 0, Ba, k, 0); SCHED; S0; SCHED; MMAQ(ai, 0, Ba, k, 2); SCHED; S1; SCHED; \
            MMAQ(ai, 1, Bb, k, 0); SCHED; S2; SCHED; MMAQ(ai, 1, Bb, k, 2); } while (0)
#define MMAK2(ai, Ba, Bb, k) do { MMAQ(ai, 0, Ba, k, 0); MMAQ(ai, 0, Ba, k, 2); MMAQ(ai, 1, Bb, k, 0); MMAQ(ai, 1, Bb, k, 2); } while (0)
#define MMA2(ai, Ba, Bb) do { __builtin_amdgcn_s_setprio(1); MMAK2(ai, Ba, Bb, 0); MMAK2(ai, Ba, Bb, 1); __builtin_amdgcn_s_setprio(0); } while (0)
#define MMA2N(ai, Ba, Bb, S0, S1, S2, S3, S4, S5) do { __builtin_amdgcn_s_setprio(1); MMAK2S(ai, Ba, Bb, 0, S0, S1, S2); SCHED; MMAK2S(ai, Ba, Bb, 1, S3, S4, S5); __builtin_amdgcn_s_setprio(0); } while (0)
#define MMA2H(ai, Ba, Bb, S0, S1, HOOK) do { __builtin_amdgcn_s_setprio(1); MMAK2S(ai, Ba, Bb, 0, S0, S1, (void)0); SCHED; HOOK; SCHED; MMAK2(ai, Ba, Bb, 1); __builtin_amdgcn_s_setprio(0); } while (0)

    const u32x4 rW = mk_desc(p.w + sB_w, W_BYTES - sB_w);
    const char* const xown = uni(p.x + (size_t)rank * CHUNK_BYTES);
    u32x4 rA = mk_desc(xown + sA_w, CHUNK_BYTES - sA_w);
    const unsigned soB0 = (unsigned)(rank * KC) * WROW;

    unsigned vSoB = (unsigned)(rank * KC + (lane & 3) * 128) * WROW;
    if (wid == 0) asm volatile("ds_write_b32 %0, %1" :: "v"((unsigned)(size_t)lds + LDS_TAB + (unsigned)(lane & 31) * 4u), "v"(vSoB) : "memory");
    constexpr bool STREAM_W = false;
    STAGE_B(0, 0, soB0); STAGE_A(0, 0, rA, 0u); STAGE_B(0, 1, soB0);
    STAGE_B(1, 0, soB0 + 64u * WROW); STAGE_A(0, 1, rA, 0u);
#pragma unroll
    for (int r = 36; r < 164; ++r) asm volatile("v_mov_b32 v%c0, 0" :: "i"(r));
    WAIT_V(10);
    unsigned cntv;
    asm volatile("ds_read_b32 %0, %1\n\ts_waitcnt lgkmcnt(0)" : "=&v"(cntv) : "v"(ldsbase + LDS_CNT + (unsigned)wid * 256u) : "memory");
    const unsigned call = (unsigned)__builtin_amdgcn_readfirstlane(cntv) / GRID + 1u;
    const size_t slot_off = WS_STAGE + (size_t)(call & 1u) * SLOT_BYTES;
    const char* const stage = uni(myws + slot_off);
    char* const pdst = uni(dws + slot_off + (size_t)rank * CHUNK_BYTES + (size_t)(pj * 16u) * AROW);
    char* const pflag = uni(dws + WS_FLAGS + (size_t)rank * 128 + pj);
    {
        const unsigned h_ = pusher ? VOFFP(lane) : CNT_VOFF(lane), po_ = PIECE_OFF(min(lane, 15)), ta_ = ldsbase + LDS_TAB + (unsigned)(lane & 31) * 4u;
        asm volatile("v_mov_b32 v244, %0\n\tv_mov_b32 v245, %1\n\tv_mov_b32 v246, %3\n\tv_mov_b32 v247, %2" :: "v"(h_), "v"(po_), "v"(ta_), "s"(call) : "memory", "v244", "v245", "v246", "v247");
    }
    if (pusher)
        asm volatile("buffer_store_byte v246, v244, %0, 0 offen\n\tbuffer_store_byte v246, v244, %0, 0 offen" :: "s"(mk_desc(psrc, 0u)) : "memory");
    if (pusher0) {
        const unsigned vp = VOFFP(lane);
        asm volatile("global_store_dwordx4 %0, v[248:251], %1 sc0 sc1\n\tglobal_store_dwordx4 %0, v[252:255], %2 sc0 sc1\n\ts_nop 1"
                     :: "v"(vp), "s"(pdst + PIECE_OFF(0)), "s"(pdst + PIECE_OFF(1)) : "memory");
    }
    STAGE_A(1, 0, rA, 128u); STAGE_B(1, 1, soB0 + 64u * WROW);
    if (pusher || pusher0) WAIT_V(10); else WAIT_V(8);
    BAR;
    LDB(B0a, 0, 0);
    if (wr == 1) BAR;

    const unsigned long long xb = (unsigned long long)(p.x + sA_w), sb = (unsigned long long)(stage + sA_w);
    auto mkA = [&](int pr, unsigned so) {
        const unsigned long long base = (pr < 4 ? xb : sb) + (((so >> 25) << 19) | ((so >> 15) & 0x300u));
        return mk_desc((const void*)base, CHUNK_BYTES - sA_w); };
    unsigned soB2 = __builtin_amdgcn_readlane(vSoB, 1);
    u32x4 rA2 = mkA(1, soB2);

#define PIECE_TAB(dst, idx) asm volatile("v_readlane_b32 %0, v245, %1" : "=s"(dst) : "s"(idx))
    const u32x4 dL = mk_desc(psrc, 16384u);
    unsigned mR = __builtin_amdgcn_readfirstlane(wid == 0 ? ((1u << 2) | (1u << 5) | (1u << 9) | (1u << 16) | (1u << 23)) : 0u);
#define SLOT(reg, c0, c1, c2, c3, dS, oS, oL) asm volatile("s_nop 4\n\tbuffer_store_dwordx4 " reg ", v244, %0, %1 offen sc0 sc1\n\ts_nop 1\n\tbuffer_load_dwordx4 " reg ", v244, %2, %3 offen" \
            :: "s"(dS), "s"(oS), "s"(dL), "s"(oL) : "memory", c0, c1, c2, c3)
    constexpr unsigned BLOCKS = 0x7F7F7F7Fu;
#define SW1(r) "v_or_b32 v249, 0x80808080, " r "\n\tv_subrev_u32 v249, %1, v249\n\tv_xor_b32 v249, v249, " r "\n\tv_xor_b32 v249, %2, v249\n\tv_or_b32 v250, v250, v249\n\t"
#define READY_MASK(R) do { unsigned long long ok_; unsigned c_ = call; asm volatile("" : "+s"(c_)); \
            const unsigned fcl = (c_ & 127u) * 0x01010101u; const unsigned fnc = ~((c_ & 255u) * 0x01010101u); \
            asm volatile("v_mov_b32 v250, 0\n\t" SW1("v252") SW1("v253") SW1("v254") SW1("v255") \
                         "v_and_b32 v250, 0x80808080, v250\n\tv_cmp_eq_u32 %0, 0, v250" : "=s"(ok_) : "s"(fcl), "s"(fnc) : "memory", "v249", "v250"); \
            R = (unsigned)ok_ & (unsigned)(ok_ >> 32) & BLOCKS; } while (0)
    unsigned used = 0, late = 0;
    auto low_bits = [](unsigned x, unsigned n) { unsigned t = x;
#pragma unroll
        for (unsigned i = 0; i < 7; ++i) t = (i < n) ? (t & (t - 1u)) : t;
        return x ^ t; };
    unsigned ndec = 4;
#define HK_PICK do { if (UNLIKELY(mR & bitu)) { \
            const unsigned nmax = ((u < 9) ? 11u : (unsigned)u + 9u) - ndec; const unsigned nmin = (u == 2) ? 3u : nmax; \
            const unsigned rest_ = BLOCKS & ~used; \
              \
            const unsigned cand_ = ((unsigned)__builtin_popcount(rest_ & ~late) >= nmax) ? (rest_ & ~late) : rest_; \
            unsigned pick = low_bits(cand_, nmax); \
            { \
                unsigned avail; READY_MASK(avail); \
                if ((avail & pick) != pick && u == 2 && (unsigned)__builtin_popcount(avail & rest_) >= nmin) pick = low_bits(avail & rest_, nmax);     \
                if ((avail & pick) != pick) { \
                    unsigned long long tt; RTC(tt); const unsigned t0 = (unsigned)tt; \
                    for (;;) { \
                        unsigned tm; \
                        asm volatile("buffer_load_dwordx4 v[252:255], v244, %0, 0 offen sc0 sc1\n\tbuffer_load_dword v248, off, %1, 0 sc1\n\ts_waitcnt vmcnt(0)" \
                                     :: "s"(mk_desc(myws + WS_FLAGS, 1024u)), "s"(mk_desc(myws + WS_TMO, 4u)) : "memory", "v252", "v253", "v254", "v255", "v248"); \
                        READY_MASK(avail); avail &= rest_; \
                        if ((avail & pick) == pick) break; \
                        asm volatile("v_mov_b32 %0, v248" : "=v"(tm) :: "memory"); \
                        RTC(tt); const unsigned dt = (unsigned)tt - t0; \
                        if (dt > PATIENCE_TICKS && (unsigned)__builtin_popcount(avail) >= nmin) {     \
                            late |= (((pick & ~avail) | ((pick & ~avail) >> 8) | ((pick & ~avail) >> 16) | ((pick & ~avail) >> 24)) & 0x7Fu) * 0x01010101u; \
                            pick = low_bits(((unsigned)__builtin_popcount(avail & ~late) >= nmin) ? (avail & ~late) : avail, nmin); break; } \
                        if ((unsigned)__builtin_amdgcn_readfirstlane(tm) == call || dt > DEADLINE_TICKS) { \
                            int lc; LANE_NOW(lc);                 \
                            if (lc == 0) asm volatile("buffer_store_dword v246, off, %0, 0 sc1\n\ts_waitcnt vmcnt(0)" :: "s"(mk_desc(myws + WS_TMO, 4u)) : "memory"); \
                            break; } \
                        __builtin_amdgcn_s_sleep(2); \
                    } \
                } \
            } \
            used |= pick; \
            int l_; LANE_NOW(l_); \
            const unsigned so_ = (unsigned)((((rank + 1 + (l_ & 7)) & 7) * KC) + ((l_ >> 3) & 3) * 128) * WROW; \
            const unsigned rk_ = __builtin_amdgcn_mbcnt_lo(pick, 0u); \
            if (l_ < 32 && ((pick >> (l_ & 31)) & 1u)) asm volatile("ds_write_b32 %0, %1" :: "v"(ldsbase + LDS_TAB + (ndec + rk_) * 4u), "v"(so_) : "memory"); \
            ndec += (unsigned)__builtin_popcount(pick); \
            if (ndec >= 11u) mR &= ~(1u << 5); \
            WAIT_L(0); } } while (0)
#define PREFETCH_W(pair, quarter) do { const unsigned so_ = __builtin_amdgcn_readlane(vSoB, pair) + (unsigned)(pm * 64 + (quarter) * 16) * WROW + (unsigned)pn * 512u; \
            int l_; LANE_NOW(l_); const unsigned vo_ = (unsigned)(l_ >> 2) * WROW + (unsigned)(l_ & 3) * 128u; \
            asm volatile("global_load_dword v249, %0, %1" :: "v"(vo_), "s"(p.w + so_) : "memory", "v249"); } while (0)
#define PREFETCH_A(pair0) do { int l_; LANE_NOW(l_); const int pr_ = (pair0); \
            const unsigned so_ = __builtin_amdgcn_readlane(vSoB, pr_); \
            const unsigned vo_ = (so_ >> 25) * CHUNK_BYTES + ((so_ >> 23) & 3u) * 256u + (unsigned)(pm * 256 + (jj >> 1) * 16 + ((l_ >> 1) & 15)) * AROW + (unsigned)(l_ & 1) * 128u; \
            asm volatile("buffer_load_dword v250, %0, %1, 0 offen" :: "v"(vo_), "s"(mk_desc(stage, (unsigned)SLOT_BYTES)) : "memory", "v250"); } while (0)
#define PHASE(LB, ab, ah, bb, bh, D0, D1, D2, D3, WAITV, MM) do { \
            LDA_H(ab, ah, 0); SCHED; D0; SCHED; LDA_H(ab, ah, 1); SCHED; D1; SCHED; LDB_H(LB, bb, bh, 0); SCHED; D2; SCHED; LDB_H(LB, bb, bh, 1); SCHED; D3; \
            WAITV; WAIT_L(0); BAR; SCHED; MM; BAR; SCHED; } while (0)
#define PAIR_BODY(WA, WB, WC, WD, MA, MB, MC, TAB) do { unsigned soB3; unsigned aof_; unsigned long long ab_; \
            PHASE(B1, 0, 0, 0, 1, DMA_B(0, 0, 0, soB2), DMA_B(0, 0, 1, soB2), DMA_A(1, 1, 0, rA, 128u), DMA_A(1, 1, 1, rA, 128u), WA, MA); \
            PHASE(B0b, 0, 1, 1, 0, DMA_A(0, 0, 0, rA2, 0u), DMA_A(0, 0, 1, rA2, 0u), DMA_B(0, 1, 0, soB2), DMA_B(0, 1, 1, soB2), WB, MB); \
            PHASE(B1, 1, 0, 1, 1, DMA_B(1, 0, 0, soB2 + 64u * WROW), DMA_B(1, 0, 1, soB2 + 64u * WROW), DMA_A(0, 1, 0, rA2, 0u), DMA_A(0, 1, 1, rA2, 0u), WC, MC); \
            PHASE(B0a, 1, 1, 0, 0, DMA_A(1, 0, 0, rA2, 128u), DMA_A(1, 0, 1, rA2, 128u), DMA_B(1, 1, 0, soB2 + 64u * WROW), DMA_B(1, 1, 1, soB2 + 64u * WROW), do { TAB; WD; } while (0), \
                  MMA2N(1, B0b, B1, do { rA = rA2; } while (0), soB3 = __builtin_amdgcn_readlane(vSoB, (u + 2) & 31), aof_ = ((soB3 >> 25) << 19) | ((soB3 >> 15) & 0x300u), \
                        ab_ = (u + 2 < 4) ? xb : sb, ab_ += aof_, rA2 = mk_desc((const void*)ab_, CHUNK_BYTES - sA_w))); \
            soB2 = soB3; } while (0)
#define TAB_REFRESH asm volatile("ds_read_b32 %0, v247" : "=v"(vSoB) :: "memory")
#define BODY_PLAIN do { PAIR_BODY(WAIT_V(8), WAIT_V(8), WAIT_V(8), WAIT_V(8), MMA2(0, B0a, B1), MMA2(1, B0a, B1), MMA2(0, B0b, B1), (void)0); } while (0)
#define BODY_PICK do { const unsigned bitu = 1u << u; \
            if (UNLIKELY(mR & bitu))        \
                asm volatile("buffer_load_dwordx4 v[252:255], v244, %0, 0 offen sc0 sc1" :: "s"(mk_desc(myws + WS_FLAGS, 1024u)) : "memory", "v252", "v253", "v254", "v255"); \
            PAIR_BODY(WAIT_V(8), WAIT_V(8), WAIT_V(8), WAIT_V(8), MMA2(0, B0a, B1), MMA2(1, B0a, B1), MMA2H(0, B0b, B1, (void)0, (void)0, HK_PICK), TAB_REFRESH); } while (0)
    auto main_part = [&](auto tag) {
    constexpr bool STREAM_W = decltype(tag)::value;
    int u = 0;
    if (pusher) {
        const u32x4 dZ = mk_desc(psrc, 0u);
#pragma clang loop unroll(disable)
        for (; u < 8; ++u) {
            const unsigned bitu = 1u << u;
            u32x4 dS, dF; unsigned oS, oL;
            PAIR_BODY(WAIT_V(10), WAIT_V(10), WAIT_V(12), WAIT_V(12),
                      MMA2H(0, B0a, B1,
                            dS = mk_desc(pdst, (0x7Fu & bitu) ? 16384u : 0u), do { PIECE_TAB(oS, 2 * u + 2); PIECE_TAB(oL, 2 * u + 4); } while (0),
                            SLOT("v[248:251]", "v248", "v249", "v250", "v251", dS, oS, oL)),
                      MMA2H(1, B0a, B1,
                            dF = mk_desc(pflag + ((u >> 1) & 3) * 32, (0xAAu & bitu) ? 1u : 0u), (void)0,
                            asm volatile("buffer_store_byte v246, v244, %0, 0 offen sc0 sc1\n\tbuffer_store_byte v246, v244, %1, 0 offen" :: "s"(dF), "s"(dZ) : "memory")),
                      MMA2H(0, B0b, B1,
                            PIECE_TAB(oS, 2 * u + 3), PIECE_TAB(oL, 2 * u + 5),
                            SLOT("v[252:255]", "v252", "v253", "v254", "v255", dS, oS, oL)),
                      TAB_REFRESH);
        }
    } else {
#pragma clang loop unroll(disable)
        for (; u < 8; ++u) BODY_PICK;
    }
#pragma clang loop unroll(disable)
    for (int seg = 0; seg < 5; ++seg) {
        const int np = (0x31661 >> (4 * seg)) & 15;
        if (seg == 4) {
#pragma clang loop unroll(disable)
            for (; u < 28; ++u) {
                PAIR_BODY(WAIT_V(8), WAIT_V(9), WAIT_V(10), WAIT_V(9),
                          MMA2H(0, B0a, B1, (void)0, (void)0, do { if (wid == 4) PREFETCH_A(u + 4); else PREFETCH_W(u + 4, wid & 3); } while (0)),
                          MMA2H(1, B0a, B1, (void)0, (void)0, asm volatile("buffer_load_dword v248, off, %0, 0 sc1" :: "s"(mk_desc(myws + WS_TMO, 4u)) : "memory", "v248")),
                          MMA2(0, B0b, B1), (void)0);
            }
        } else if (seg > 0) {
            BODY_PICK;
            ++u;
        }
#pragma clang loop unroll(disable)
        for (int i = 0; i < np; ++i, ++u) BODY_PLAIN;
    }

    int tz = threadIdx.x; asm volatile("" : "+v"(tz));
    const int l2 = tz & 63, fr2 = l2 & 15, fq2 = l2 >> 4;
    const unsigned voffO = (unsigned)fr2 * WROW + (unsigned)((fq2 & 1) * 32 + (fq2 >> 1) * 16);
    char* const obase = uni(p.out + (size_t)(pm * 256 + wr * 64) * WROW + (size_t)pn * 512 + wc * 64);
    bool bad;
#define CVT(dst, r) asm volatile("v_cvt_pk_bf16_f32 %0, v%c1, v%c2" : "=v"(dst) : "i"(r), "i"((r) + 1))
#define OSTORE(o, rowp, bj) asm volatile("global_store_dwordx4 %0, %1, %2 offset:%c3 sc1 nt\n\ts_nop 1" :: "v"(voffO), "v"(o), "s"(rowp), "i"((bj) * 256) : "memory")
#define EPI(ai) do { if (UNLIKELY(bad)) { const u32x4 q_ = {0x7fc07fc0u, 0x7fc07fc0u, 0x7fc07fc0u, 0x7fc07fc0u}; \
            _Pragma("unroll") for (int m = 0; m < 4; ++m) { char* const rowp = obase + (size_t)((ai) * 128 + m * 16) * WROW; OSTORE(q_, rowp, 0); OSTORE(q_, rowp, 1); } \
        } else { _Pragma("unroll") for (int m = 0; m < 4; ++m) { \
            char* const rowp = obase + (size_t)((ai) * 128 + m * 16) * WROW; \
            _Pragma("unroll") for (int bj = 0; bj < 2; ++bj) { \
                unsigned a0; unsigned a1; unsigned b0; unsigned b1; \
                CVT(a0, ACC(ai, bj, m, 0)); CVT(a1, ACC(ai, bj, m, 0) + 2); CVT(b0, ACC(ai, bj, m, 1)); CVT(b1, ACC(ai, bj, m, 1) + 2); \
                a0 = relu_pk(a0); a1 = relu_pk(a1); b0 = relu_pk(b0); b1 = relu_pk(b1); \
                auto r0 = __builtin_amdgcn_permlane16_swap(a0, b0, false, false); \
                auto r1 = __builtin_amdgcn_permlane16_swap(a1, b1, false, false); \
                u32x4 o; o.x = r0[0]; o.y = r1[0]; o.z = r0[1]; o.w = r1[1]; \
                OSTORE(o, rowp, bj); } } } } while (0)
#define PHASE_L(L1, L2, S2, WAITV, MM) do { L1; L2; SCHED; S2; WAITV; WAIT_L(0); BAR; SCHED; MM; BAR; SCHED; } while (0)
    PHASE_L(LDA(0, 0), LDB(B1, 0, 1), STAGE_A(1, 1, rA, 128u), WAIT_V(6), MMA2(0, B0a, B1));
    { unsigned tm; asm volatile("v_mov_b32 %0, v248" : "=v"(tm) :: "memory"); bad = ((unsigned)__builtin_amdgcn_readfirstlane(tm) == call); }
    PHASE_L(LDA(0, 1), LDB(B0b, 1, 0), (void)0, WAIT_V(2), MMA2(1, B0a, B1));
    PHASE_L(LDA(1, 0), LDB(B1, 1, 1), (void)0, WAIT_V(0), MMA2(0, B0b, B1));
    LDA(1, 1); SCHED;
    asm volatile("s_nop 15" ::: "memory");
    EPI(0);
    WAIT_L(0); BAR; SCHED;
    MMA2(1, B0b, B1);
    if (wr == 0) BAR;
    SCHED;
    asm volatile("s_nop 15\n\ts_nop 15" ::: "memory");
    EPI(1);
    };
    if (call == 1u) main_part(BoolTag<true>{}); else main_part(BoolTag<false>{});
    if (tid == 0) __hip_atomic_fetch_add((GAS unsigned*)(myws + WS_CNT), 1u, __ATOMIC_RELAXED, __HIP_MEMORY_SCOPE_AGENT);
}

extern "C" void kernel_launch(int rank, int nranks, ncclComm_t comm,
                              hipStream_t stream,
                              void* const* d_in, const int* in_sizes, int n_in,
                              void* d_out, int out_size,
                              const int* in_dtypes, int out_dtype,
                              void* const* d_out_all,
                              void* const* d_ws_all, size_t ws_bytes) {
    (void)comm; (void)in_dtypes; (void)out_dtype; (void)d_out_all;
    if (nranks != NR || n_in < 2 || rank < 0 || rank >= NR || ws_bytes < WS_NEED ||
        in_sizes[0] != KDIM * KC || in_sizes[1] != KDIM * NDIM || out_size != MLOC * NDIM) {
        fprintf(stderr, "kernel_launch: unexpected problem shape\n");
        return;
    }
    static bool first = true;
    if (first) {
        HIP_CHECK(hipFuncSetAttribute((const void*)a2a_gemm_kernel, hipFuncAttributeMaxDynamicSharedMemorySize, LDS_BYTES));
        first = false;
    }
    Params p;
    p.x = (const char*)d_in[0];
    p.w = (const char*)d_in[1];
    p.out = (char*)d_out;
    for (int i = 0; i < NR; ++i) p.ws[i] = (char*)d_ws_all[i];
    p.rank = rank;
    a2a_gemm_kernel<<<GRID, 512, LDS_BYTES, stream>>>(p);
    HIP_CHECK(hipGetLastError());
}
```
